```python
import numpy as np

kernel_name = "gpu_gemm_ar_m4096_k4096_n32768_f32_none"


ABSMAX_RTOL = 0.001

COMPUTE_DTYPE = "float32"
EPILOGUE = "none"
IN_QUANT = None
OUT_QUANT = None
IN_DTYPES = {"x": "float32", "w_mat": "float32"}
WORLD_SIZE = 8

def setup_inputs():

    x = np.random.randn(4096, 4096).astype(np.float32)
    w_mat = (np.random.randn(4096, 32768) / np.sqrt(4096)).astype(np.float32)
    return {"x": x, "w_mat": w_mat}

def _epilogue(y):
    return y

def reference(x, w_mat):
    y = (x @ w_mat).astype(np.float32)
    return _epilogue(y)
```

```cpp
#include <hip/hip_runtime.h>
#include <rccl/rccl.h>
#include <cstdio>

#define HIP_CHECK(expr) do { hipError_t _e = (expr); if (_e != hipSuccess) { \
    fprintf(stderr, "HIP error %s at %s:%d\n", hipGetErrorString(_e), __FILE__, __LINE__); \
    return; } } while (0)

namespace {
#define LAS __attribute__((address_space(3)))
typedef _Float16 h8 __attribute__((ext_vector_type(8)));
typedef _Float16 h2 __attribute__((ext_vector_type(2)));
typedef float f32x4 __attribute__((ext_vector_type(4)));
typedef unsigned u32x4 __attribute__((ext_vector_type(4)));
typedef __amdgpu_buffer_rsrc_t rsrc_t;

constexpr int NR = 8, M = 4096, N = 32768, KL = 512, NKT = 64, GRID = 256;
constexpr unsigned HTB = 16384, KTB = 32768, PANEL = NKT * KTB;
constexpr unsigned A_BYTES = (M / 256) * PANEL, B_BYTES = (N / 256) * PANEL, BUF_BYTES = A_BYTES + B_BYTES;
constexpr unsigned OUT_BYTES = 512u << 20, X_BYTES = M * KL * 4, W_BYTES = KL * N * 4;
constexpr size_t CTRL_BYTES = 1u << 20, WS_NEED = CTRL_BYTES + 2 * (size_t)BUF_BYTES;
constexpr int W_DONE = 0, W_TMO = 32, W_CNT = 8 * 32, W_FWD = W_CNT + 2 * 64 * 8 * 32;
constexpr unsigned CNT_PER_CALL = 240, FWD_PER_CALL = 128;
constexpr unsigned long long DEADLINE = 50000000ull;
constexpr int LDS_BYTES = 8 * HTB + 1024;
constexpr int AUX_SYS = 17, AUX_NT = 2;

struct Params { const float* x; const float* w; float* out[NR]; unsigned char* ws[NR]; int rank; };

__device__ __forceinline__ rsrc_t mk(const void* p, unsigned bytes) { return __builtin_amdgcn_make_buffer_rsrc((void*)p, 0, bytes, 0x00020000); }
__device__ __forceinline__ unsigned pk(float a, float b) { h2 h; h.x = (_Float16)a; h.y = (_Float16)b; return __builtin_bit_cast(unsigned, h); }
#define WAIT_V(n) asm volatile("s_waitcnt vmcnt(" #n ")" ::: "memory")
#define WAIT_L(n) asm volatile("s_waitcnt lgkmcnt(" #n ")" ::: "memory")
#define BAR __builtin_amdgcn_s_barrier()
#define SCHED __builtin_amdgcn_sched_barrier(0)

__device__ __forceinline__ bool wait8(unsigned* base, unsigned thr, int skip, unsigned* tmo, unsigned call, int lane) {
    const unsigned long long t0 = __builtin_amdgcn_s_memrealtime();
    for (;;) {
        const unsigned v = __hip_atomic_load(base + (lane & 7) * 32, __ATOMIC_RELAXED, __HIP_MEMORY_SCOPE_SYSTEM);
        if (__all(v >= thr || (lane & 7) == skip)) return true;
        if (__builtin_amdgcn_readfirstlane(__hip_atomic_load(tmo, __ATOMIC_RELAXED, __HIP_MEMORY_SCOPE_AGENT)) == call
            || __builtin_amdgcn_s_memrealtime() - t0 > DEADLINE) {
            __hip_atomic_store(tmo, call, __ATOMIC_RELAXED, __HIP_MEMORY_SCOPE_AGENT);
            return false;
        }
        __builtin_amdgcn_s_sleep(4);
    }
}

struct Unit { unsigned src, dst; int isA, dest; };
__device__ __forceinline__ Unit unit_a(int pm, int s, int sub, int rank) {
    Unit u; u.isA = 1; u.dest = -1; u.src = (unsigned)(pm * 256 + sub * 32) * (KL * 4) + s * 256;
    u.dst = pm * PANEL + (s * 8 + rank) * KTB + sub * 4096; return u; }
__device__ __forceinline__ Unit unit_b(int pn, int s, int sub, int rank) {
    Unit u; u.isA = 0; u.dest = pn >= 120 ? pn - 120 : -1; u.src = (unsigned)(s * 64 + (sub & 1) * 32) * (N * 4) + pn * 1024 + (sub >> 1) * 256;
    u.dst = A_BYTES + pn * PANEL + (s * 8 + rank) * KTB + (sub >> 1) * 4096 + (sub & 1) * 1024; return u; }
__device__ __forceinline__ void unit_load(const Unit& u, rsrc_t rx, rsrc_t rw, int lane, f32x4 (&v)[8]) {
    if (u.isA) {
        const unsigned o = u.src + (lane & 15) * (KL * 4) + (lane >> 4) * 32;
#pragma unroll
        for (int i = 0; i < 8; ++i) v[i] = __builtin_bit_cast(f32x4, __builtin_amdgcn_raw_buffer_load_b128(rx, o + (i >> 2) * 16 * (KL * 4) + ((i >> 1) & 1) * 128 + (i & 1) * 16, 0, AUX_NT));
    } else {
        const unsigned o = u.src + (lane >> 4) * 8 * (N * 4) + (lane & 15) * 16;
#pragma unroll
        for (int j = 0; j < 8; ++j) v[j] = __builtin_bit_cast(f32x4, __builtin_amdgcn_raw_buffer_load_b128(rw, o + j * (N * 4), 0, AUX_NT));
    }
}
__device__ __forceinline__ void unit_cvt(const Unit& u, const f32x4 (&v)[8], u32x4 (&h)[4]) {
#define F(i, c) v[i][c]
    if (u.isA) {
#pragma unroll
        for (int i = 0; i < 4; ++i) { h[i][0] = pk(F(2 * i, 0), F(2 * i, 1)); h[i][1] = pk(F(2 * i, 2), F(2 * i, 3)); h[i][2] = pk(F(2 * i + 1, 0), F(2 * i + 1, 1)); h[i][3] = pk(F(2 * i + 1, 2), F(2 * i + 1, 3)); }
    } else {
#pragma unroll
        for (int i = 0; i < 4; ++i) { h[i][0] = pk(F(0, i), F(1, i)); h[i][1] = pk(F(2, i), F(3, i)); h[i][2] = pk(F(4, i), F(5, i)); h[i][3] = pk(F(6, i), F(7, i)); }
    }
#undef F
}
template <int AUX>
__device__ __forceinline__ void unit_store(const Unit& u, const Params& p, size_t bufoff, int rot, int lane, const u32x4 (&h)[4]) {
    const unsigned o = u.dst + lane * 16, s1 = u.isA ? 1024u : 2048u, s2 = u.isA ? 2048u : 16384u;
#pragma unroll 1
    for (int q = 0; q < NR; ++q) {
        const int d = (q + rot) & 7;
        if (u.dest >= 0 && u.dest != d) continue;
        const rsrc_t r = mk(p.ws[d] + bufoff, BUF_BYTES);
        __builtin_amdgcn_raw_buffer_store_b128(h[0], r, o, 0, AUX);
        __builtin_amdgcn_raw_buffer_store_b128(h[1], r, o + s1, 0, AUX);
        __builtin_amdgcn_raw_buffer_store_b128(h[2], r, o + s2, 0, AUX);
        __builtin_amdgcn_raw_buffer_store_b128(h[3], r, o + s1 + s2, 0, AUX);
    }
}
__device__ __forceinline__ void signal8(const Params& p, int word, unsigned add, int lane) {
    if (lane < NR) {
        unsigned char* b = p.ws[0];
#pragma unroll
        for (int d = 1; d < NR; ++d) b = lane == d ? p.ws[d] : b;
        __hip_atomic_fetch_add((unsigned*)b + word, add, __ATOMIC_RELAXED, __HIP_MEMORY_SCOPE_SYSTEM);
    }
}
__device__ __forceinline__ bool p1_decode(int u, int& sigma, int& pn, int& np) {
    int b, r;
    if (u < 384) { b = 1 + (u >> 7); r = u & 127; np = 16; }
    else if (u < 576) { b = 4; r = u - 384; np = 24; }
    else if (u < 832) { b = 5 + ((u - 576) >> 7); r = (u - 576) & 127; np = 16; }
    else if (u < 912) { b = 7; r = u - 832; np = 10; }
    else return false;
    const int s = r / np, i = r - s * np;
    sigma = b * 8 + s; pn = i < 16 ? b * 16 - 2 + i : 120 + (i - 16);
    return true;
}

struct Tile { int pm, pn, blk, split; };
__device__ __forceinline__ bool slot(int j, int g, int x, int rank, Tile& t) {
    int e;
    if (j == 0) { if (g < 3) return false; e = g - 3; }
    else if (j <= 4) e = 28 + 31 * (j - 1) + g;
    else if (j == 5) { if (g >= 29) { t.pm = 4 * (x & 3) + 2 * (x >> 2) + (g - 29); t.pn = 120 + rank; t.blk = 4; t.split = 1; return true; } e = 152 + g; }
    else if (j == 6) { if (g >= 29) return false; e = 182 + g; }
    else { if (g == 30) return false; e = g == 29 ? 181 : 211 + g; }
    const int np = e >> 2;
    t.pm = 4 * (x & 3) + (e & 3); t.pn = 2 * np + (x >> 2); t.blk = (np + 1) >> 3; t.split = 0;
    return true;
}
__device__ __forceinline__ bool next_slot(int& j, int g, int x, int rank, Tile& t) {
    for (++j; j < 8; ++j) if (slot(j, g, x, rank, t)) return true;
    return false;
}

#define SA(b, h) (((b) * 2 + (h)) * HTB)
#define SB(b, h) ((4 + (b) * 2 + (h)) * HTB)
#define STAGE(bufoff, soff) do { \
    __builtin_amdgcn_raw_ptr_buffer_load_lds(rAB, (LAS void*)(lds + (bufoff) + ldsw), 16, voff, (soff), 0, AUX_SYS); \
    __builtin_amdgcn_raw_ptr_buffer_load_lds(rAB, (LAS void*)(lds + (bufoff) + ldsw), 16, voff, (soff), 1024, AUX_SYS); } while (0)
#define LDA(dst, b, h) do { _Pragma("unroll") for (int m = 0; m < 4; ++m) _Pragma("unroll") for (int k = 0; k < 2; ++k) dst[m][k] = *(const LAS h8*)(lds + SA(b, h) + aoff + m * 2048 + k * 1024); } while (0)
#define LDB(dst, b, h) do { _Pragma("unroll") for (int n = 0; n < 2; ++n) _Pragma("unroll") for (int k = 0; k < 2; ++k) dst[n][k] = *(const LAS h8*)(lds + SB(b, h) + boff + n * 2048 + k * 1024); } while (0)
#define MMA(ai, bj, At, Bt) do { __builtin_amdgcn_s_setprio(1); _Pragma("unroll") for (int m = 0; m < 4; ++m) _Pragma("unroll") for (int n = 0; n < 2; ++n) _Pragma("unroll") for (int k = 0; k < 2; ++k) \
    acc[ai][bj][m][n] = __builtin_amdgcn_mfma_f32_16x16x32_f16(At[m][k], Bt[n][k], acc[ai][bj][m][n], 0, 0, 0); __builtin_amdgcn_s_setprio(0); } while (0)

__global__ __launch_bounds__(512, 2) void fused_gemm(Params p) {
    extern __shared__ __attribute__((aligned(16))) unsigned char shm[];
    LAS unsigned char* lds = (LAS unsigned char*)shm;
    const int tid = threadIdx.x, wid = __builtin_amdgcn_readfirstlane(tid >> 6), lane = tid & 63;
    const int x = blockIdx.x & 7, c = blockIdx.x >> 3, rank = p.rank;
    unsigned* ctrl = (unsigned*)p.ws[rank];
    const unsigned call = __builtin_amdgcn_readfirstlane(__hip_atomic_load(ctrl + W_DONE, __ATOMIC_RELAXED, __HIP_MEMORY_SCOPE_AGENT)) / GRID + 1;
    const unsigned par = call & 1, ncp = (call + 1) >> 1;
    const size_t bufoff = CTRL_BYTES + (size_t)par * BUF_BYTES;
    const int cntw = W_CNT + par * 64 * 8 * 32;

    if (c < 4) {
        const rsrc_t rx = mk(p.x, X_BYTES), rw = mk(p.w, W_BYTES);
        f32x4 v[8]; u32x4 h[4];
        const int q0 = c * 8 + x;
        if (q0 < 30) {
            Unit u = q0 < 16 ? unit_a(q0, 0, wid, rank) : unit_b(q0 - 16, 0, wid, rank);
            unit_load(u, rx, rw, lane, v);
            for (int s = 0; s < 8; ++s) {
                unit_cvt(u, v, h);
                Unit un = q0 < 16 ? unit_a(q0, (s + 1) & 7, wid, rank) : unit_b(q0 - 16, (s + 1) & 7, wid, rank);
                unit_load(un, rx, rw, lane, v);
                unit_store<AUX_SYS>(u, p, bufoff, rank + 1 + wid + q0, lane, h);
                WAIT_V(0);
                signal8(p, cntw + s * 256 + rank * 32, 1u, lane);
                u = un;
            }
        }
        if (c == 0) {
            LAS unsigned* sw = (LAS unsigned*)(lds + 8 * HTB);
            if (tid < 128) sw[tid] = 0;
            __syncthreads();
            int sg, pn, np, sgn, pnn, npn;
            int ui = __builtin_amdgcn_readfirstlane(lane == 0 ? __hip_atomic_fetch_add(sw, 1u, __ATOMIC_RELAXED, __HIP_MEMORY_SCOPE_WORKGROUP) : 0u);
            bool ok = p1_decode(ui, sg, pn, np);
            Unit u = unit_b(ok ? pn : 0, sg & 7, x, rank);
            unit_load(u, rx, rw, lane, v);
            while (ok) {
                unit_cvt(u, v, h);
                ui = __builtin_amdgcn_readfirstlane(lane == 0 ? __hip_atomic_fetch_add(sw, 1u, __ATOMIC_RELAXED, __HIP_MEMORY_SCOPE_WORKGROUP) : 0u);
                const bool okn = p1_decode(ui, sgn, pnn, npn);
                Unit un = unit_b(okn ? pnn : 0, okn ? sgn & 7 : 0, x, rank);
                unit_load(un, rx, rw, lane, v);
                unit_store<AUX_NT>(u, p, bufoff, rank + 1 + wid + x, lane, h);
                WAIT_V(0);
                const int fin = __builtin_amdgcn_readfirstlane(lane == 0 ? __hip_atomic_fetch_add(sw + 8 + sg, 1u, __ATOMIC_RELAXED, __HIP_MEMORY_SCOPE_WORKGROUP) : 0u);
                if (fin == np - 1) {
                    __builtin_amdgcn_fence(__ATOMIC_RELEASE, "");
                    WAIT_V(0);
                    signal8(p, cntw + sg * 256 + rank * 32, CNT_PER_CALL / 8, lane);
                }
                u = un; sg = sgn; np = npn; ok = okn;
            }
            if (x == 0 && wid == 0) {
                if (!wait8(ctrl + W_FWD + par * 8 * 32, FWD_PER_CALL * ncp, rank, ctrl + W_TMO, call, lane) && lane == 0) p.out[rank][0] = __builtin_nanf("");
                __builtin_amdgcn_fence(__ATOMIC_ACQUIRE, "");
            }
        }
    }
    if (c != 0) {
        const int g = c - 1, wr = wid >> 2, wc = wid & 3, fr = lane & 15, fq = lane >> 4;
        const rsrc_t rAB = mk(p.ws[rank] + bufoff, BUF_BYTES);
        const unsigned voff = wid * 2048 + lane * 16, ldsw = wid * 2048;
        const int aoff = wr * 8192 + lane * 16, boff = wc * 4096 + lane * 16;
        const unsigned cvoff = (unsigned)(wr * 64 + 4 * fq) * (N * 4) + (wc * 64 + 4 * fr) * 4;
        const unsigned thr = CNT_PER_CALL * ncp;
        unsigned* const cnt = ctrl + cntw;
        const rsrc_t rCt = mk(ctrl, (unsigned)CTRL_BYTES);
        int j = -1; Tile cur, nxt;
        next_slot(j, g, x, rank, cur);
        bool bad = false;
        if (wid == 0) {
            if (g < 3) for (int s = 0; s < 8; ++s) bad |= !wait8(cnt + s * 256, thr, -1, ctrl + W_TMO, call, lane);
            bad |= !wait8(cnt + cur.blk * 8 * 256, thr, -1, ctrl + W_TMO, call, lane);
        }
        asm volatile("" ::: "memory"); BAR; asm volatile("" ::: "memory");
        f32x4 acc[2][2][4][2];
#pragma unroll
        for (int a = 0; a < 2; ++a)
#pragma unroll
            for (int b = 0; b < 2; ++b)
#pragma unroll
                for (int m = 0; m < 4; ++m)
#pragma unroll
                    for (int n = 0; n < 2; ++n) acc[a][b][m][n] = (f32x4){0.f, 0.f, 0.f, 0.f};
        h8 At[4][2], B0[2][2], B1[2][2];
        unsigned cA = cur.pm * PANEL, cB = A_BYTES + cur.pn * PANEL;
        STAGE(SB(0, 0), cB); STAGE(SB(0, 1), cB + HTB); STAGE(SA(0, 0), cA); STAGE(SA(0, 1), cA + HTB);
        if (wr == 1) BAR;
        WAIT_V(2); BAR;
        STAGE(SB(1, 0), cB + KTB); STAGE(SA(1, 0), cA + KTB); STAGE(SB(1, 1), cB + HTB + KTB);
        WAIT_V(6); BAR;
        for (;;) {
            const bool has_next = next_slot(j, g, x, rank, nxt);
            const unsigned nA = has_next ? nxt.pm * PANEL : cA, nB = has_next ? A_BYTES + nxt.pn * PANEL : cB;
            const int nblk = has_next ? nxt.blk : cur.blk;
            for (int t = 0; t < NKT; t += 2) {
                const bool last = (t == NKT - 2);
                const unsigned a1 = cA + (t + 1) * KTB;
                const unsigned a2 = last ? nA : cA + (t + 2) * KTB, b2 = last ? nB : cB + (t + 2) * KTB;
                const unsigned a3 = a2 + KTB, b3 = b2 + KTB;
                if (wid == 0 && (t & 4)) {
                    const int sg = t >= NKT - 4 ? nblk * 8 : cur.blk * 8 + (t >> 3) + 1;
                    if (!(t & 2)) __builtin_amdgcn_raw_ptr_buffer_load_lds(rCt, (LAS void*)(lds + 8 * HTB), 4, (lane & 7) * 128, (cntw + sg * 256) * 4, 0, AUX_SYS);
                    else { const unsigned fval = *(volatile LAS unsigned*)(lds + 8 * HTB + lane * 4);
                           if (!__all(fval >= thr)) bad |= !wait8(cnt + sg * 256, thr, -1, ctrl + W_TMO, call, lane); asm volatile("" ::: "memory"); }
                }
                LDB(B0, 0, 0); LDB(B1, 0, 1); SCHED; LDA(At, 0, 0); STAGE(SA(1, 1), a1 + HTB);
                WAIT_V(8); WAIT_L(0); BAR; MMA(0, 0, At, B0); MMA(0, 1, At, B1); BAR; SCHED;
                LDA(At, 0, 1); STAGE(SB(0, 0), b2); STAGE(SB(0, 1), b2 + HTB); STAGE(SA(0, 0), a2);
                WAIT_V(8); WAIT_L(0); BAR; MMA(1, 0, At, B0); MMA(1, 1, At, B1); BAR; SCHED;
                LDB(B0, 1, 0); LDB(B1, 1, 1); SCHED; LDA(At, 1, 0); STAGE(SA(0, 1), a2 + HTB);
                WAIT_V(8); WAIT_L(0); BAR; MMA(0, 0, At, B0); MMA(0, 1, At, B1); BAR; SCHED;
                LDA(At, 1, 1); STAGE(SB(1, 0), b3); STAGE(SB(1, 1), b3 + HTB); STAGE(SA(1, 0), a3);
                WAIT_V(8); WAIT_L(0); BAR; MMA(1, 0, At, B0); MMA(1, 1, At, B1); BAR; SCHED;
            }
            if (bad) acc[0][0][0][0][0] = __builtin_nanf("");
            const unsigned cb = cvoff + (unsigned)cur.pm * 256u * (N * 4) + (unsigned)cur.pn * 1024u;
            if (!cur.split) {
                const rsrc_t rC = mk(p.out[rank], OUT_BYTES);
#pragma unroll
                for (int ai = 0; ai < 2; ++ai)
#pragma unroll
                    for (int m = 0; m < 4; ++m)
#pragma unroll
                        for (int q = 0; q < 4; ++q) {
                            const f32x4 o = {acc[ai][0][m][0][q], acc[ai][0][m][1][q], acc[ai][1][m][0][q], acc[ai][1][m][1][q]};
                            __builtin_amdgcn_raw_buffer_store_b128(__builtin_bit_cast(u32x4, o), rC, cb + (unsigned)(ai * 128 + m * 16 + q) * (N * 4), 0, 0);
                        }
            } else {
                rsrc_t rC[NR];
#pragma unroll
                for (int dd = 0; dd < NR; ++dd) rC[dd] = mk(p.out[(dd + rank + 1 + wid + x) & 7], OUT_BYTES);
#pragma unroll
                for (int ai = 0; ai < 2; ++ai)
#pragma unroll
                    for (int m = 0; m < 4; ++m)
#pragma unroll
                        for (int q = 0; q < 4; ++q) {
                            const f32x4 o = {acc[ai][0][m][0][q], acc[ai][0][m][1][q], acc[ai][1][m][0][q], acc[ai][1][m][1][q]};
#pragma unroll
                            for (int dd = 0; dd < NR; ++dd)
                                __builtin_amdgcn_raw_buffer_store_b128(__builtin_bit_cast(u32x4, o), rC[dd], cb + (unsigned)(ai * 128 + m * 16 + q) * (N * 4), 0, AUX_SYS);
                            SCHED;
                        }
                WAIT_V(0);
                signal8(p, W_FWD + par * 8 * 32 + rank * 32, 1u, lane);
            }
            if (!has_next) break;
#pragma unroll
            for (int a = 0; a < 2; ++a)
#pragma unroll
                for (int b = 0; b < 2; ++b)
#pragma unroll
                    for (int m = 0; m < 4; ++m)
#pragma unroll
                        for (int n = 0; n < 2; ++n) acc[a][b][m][n] = (f32x4){0.f, 0.f, 0.f, 0.f};
            cur = nxt; cA = nA; cB = nB;
        }
        WAIT_V(0);
        if (wr == 0) BAR;
    }
    BAR;
    if (tid == 0) __hip_atomic_fetch_add(ctrl + W_DONE, 1u, __ATOMIC_RELAXED, __HIP_MEMORY_SCOPE_AGENT);
}
}

extern "C" void kernel_launch(int rank, int nranks, ncclComm_t comm, hipStream_t stream,
                              void* const* d_in, const int* in_sizes, int n_in,
                              void* d_out, int out_size, const int* in_dtypes, int out_dtype,
                              void* const* d_out_all, void* const* d_ws_all, size_t ws_bytes) {
    if (nranks != NR || (unsigned)rank >= (unsigned)NR || n_in != 2 || in_sizes[0] != M * KL || in_sizes[1] != KL * N
        || out_size != M * N || ws_bytes < WS_NEED) { fprintf(stderr, "unexpected problem shape\n"); return; }
    static bool init = false;
    if (!init) { HIP_CHECK(hipFuncSetAttribute((const void*)fused_gemm, hipFuncAttributeMaxDynamicSharedMemorySize, LDS_BYTES)); init = true; }
    Params p;
    p.x = (const float*)d_in[0]; p.w = (const float*)d_in[1]; p.rank = rank;
    for (int r = 0; r < NR; ++r) { p.out[r] = (float*)d_out_all[r]; p.ws[r] = (unsigned char*)d_ws_all[r]; }
    hipLaunchKernelGGL(fused_gemm, dim3(GRID), dim3(512), LDS_BYTES, stream, p);
    HIP_CHECK(hipGetLastError());
}
```

```cpp
#include <hip/hip_runtime.h>
#include <rccl/rccl.h>
#include <cstdio>

#define HIP_CHECK(expr) do { hipError_t _e = (expr); if (_e != hipSuccess) { \
    fprintf(stderr, "HIP error %s at %s:%d\n", hipGetErrorString(_e), __FILE__, __LINE__); \
    return; } } while (0)

namespace {
#define LAS __attribute__((address_space(3)))
typedef _Float16 h8 __attribute__((ext_vector_type(8)));
typedef _Float16 h2 __attribute__((ext_vector_type(2)));
typedef float f32x4 __attribute__((ext_vector_type(4)));
typedef unsigned u32x4 __attribute__((ext_vector_type(4)));
typedef __amdgpu_buffer_rsrc_t rsrc_t;

constexpr int NR = 8, M = 4096, N = 32768, KL = 512, NKT = 64, GRID = 256;
constexpr unsigned HTB = 16384, KTB = 32768, PANEL = NKT * KTB;
constexpr unsigned A_BYTES = (M / 256) * PANEL, B_BYTES = (N / 256) * PANEL, BUF_BYTES = A_BYTES + B_BYTES;
constexpr unsigned OUT_BYTES = 512u << 20, X_BYTES = M * KL * 4, W_BYTES = KL * N * 4;
constexpr size_t CTRL_BYTES = 1u << 20, WS_NEED = CTRL_BYTES + 2 * (size_t)BUF_BYTES;
constexpr int W_DONE = 0, W_TMO = 32, W_PAR = 64, PAR_WORDS = 160 * 32;
constexpr int P_TICKET = 0, P_TILE = 32, P_FWD = 64, P_PRE = 10 * 32, P_CNT = 16 * 32;
constexpr unsigned CNT_PER_CALL = 64, TILE_PER_CALL = 160, FWD_PER_CALL = 1280;
constexpr unsigned long long DEADLINE = 50000000ull;
constexpr int LDS_BYTES = 8 * HTB + 1024;
constexpr int AUX_SYS = 17, AUX_NT = 2;
#define PUSH_E 1
#define PUSH_O 3
#define POLL_SLEEP 64
#define CONV_GAP 800
#define ODD_FIRST 0.0
#define PUSH_WAVES 2
constexpr int GE = 32 - PUSH_E, GO = 32 - PUSH_O, RJ = GE + GO, NPN = 118, SPLIT_COL = 77;
constexpr int PRE_WGS = 4 * RJ, NPUSH = 4 * (PUSH_E + PUSH_O);
constexpr int NBLK = (16 + NPN) * 8, U_FWD = (NBLK - NPUSH) * 8, U_END = U_FWD + FWD_PER_CALL;
static_assert(GE >= GO && RJ == 60, "role split");

struct Params { const float* x; const float* w; float* out[NR]; unsigned char* ws[NR]; int rank; int nbs; unsigned blk[NBLK / 2]; };

__device__ __forceinline__ rsrc_t mk(const void* p, unsigned bytes) { return __builtin_amdgcn_make_buffer_rsrc((void*)p, 0, bytes, 0x00020000); }
__device__ __forceinline__ unsigned pk(float a, float b) { h2 h; h.x = (_Float16)a; h.y = (_Float16)b; return __builtin_bit_cast(unsigned, h); }
#define WAIT_V(n) asm volatile("s_waitcnt vmcnt(" #n ")" ::: "memory")
#define WAIT_L(n) asm volatile("s_waitcnt lgkmcnt(" #n ")" ::: "memory")
#define BAR __builtin_amdgcn_s_barrier()
#define SCHED __builtin_amdgcn_sched_barrier(0)
#define STAMP(i) do {} while (0)
#define STAMPV(i, v) do {} while (0)

__device__ __forceinline__ unsigned long long wait_mask(unsigned* word, unsigned thr, unsigned long long need, unsigned* tmo, unsigned call, int slow = 0) {
    const unsigned long long t0 = __builtin_amdgcn_s_memrealtime();
    for (unsigned it = 1;; ++it) {
        const unsigned long long got = __ballot(__hip_atomic_load(word, __ATOMIC_RELAXED, __HIP_MEMORY_SCOPE_SYSTEM) >= thr);
        if ((got & need) == need) return got;
        if ((it & 63) == 0 && (__builtin_amdgcn_readfirstlane(__hip_atomic_load(tmo, __ATOMIC_RELAXED, __HIP_MEMORY_SCOPE_AGENT)) == call
            || __builtin_amdgcn_s_memrealtime() - t0 > DEADLINE)) {
            __hip_atomic_store(tmo, call, __ATOMIC_RELAXED, __HIP_MEMORY_SCOPE_AGENT);
            return 0;
        }
        if (slow) __builtin_amdgcn_s_sleep(POLL_SLEEP); else __builtin_amdgcn_s_sleep(4);
    }
}

__device__ __forceinline__ void conv_load(const Params& p, int it, int s, int sub, int lane, f32x4 (&v)[8]) {
    if (it < 16) {
        const rsrc_t r = mk(p.x, X_BYTES);
        const unsigned a = (unsigned)(it * 256 + sub * 32 + (lane & 15)) * (KL * 4) + s * 256 + (lane >> 4) * 32;
#pragma unroll
        for (int i = 0; i < 8; ++i) v[i] = __builtin_bit_cast(f32x4, __builtin_amdgcn_raw_buffer_load_b128(r, a + (i >> 2) * 16 * (KL * 4) + ((i >> 1) & 1) * 128 + (i & 1) * 16, 0, AUX_NT));
    } else {
        const rsrc_t r = mk(p.w, W_BYTES);
        const unsigned a = (unsigned)(s * 64 + (sub & 1) * 32 + (lane >> 4) * 8) * (N * 4) + (it - 16) * 1024 + (sub >> 1) * 256 + (lane & 15) * 16;
#pragma unroll
        for (int j = 0; j < 8; ++j) v[j] = __builtin_bit_cast(f32x4, __builtin_amdgcn_raw_buffer_load_b128(r, a + j * (N * 4), 0, AUX_NT));
    }
}
__device__ __forceinline__ void conv_store(const Params& p, int it, int s, int sub, int rank, size_t bufoff, int dmask, int lane, const f32x4 (&v)[8]) {
    u32x4 h[4];
    unsigned o = it * PANEL + (s * 8 + rank) * KTB + lane * 16, s1, s2;
    if (it < 16) {
#pragma unroll
        for (int i = 0; i < 4; ++i) { h[i][0] = pk(v[2 * i][0], v[2 * i][1]); h[i][1] = pk(v[2 * i][2], v[2 * i][3]); h[i][2] = pk(v[2 * i + 1][0], v[2 * i + 1][1]); h[i][3] = pk(v[2 * i + 1][2], v[2 * i + 1][3]); }
        o += sub * 4096; s1 = 1024; s2 = 2048;
    } else {
#pragma unroll
        for (int i = 0; i < 4; ++i) { h[i][0] = pk(v[0][i], v[1][i]); h[i][1] = pk(v[2][i], v[3][i]); h[i][2] = pk(v[4][i], v[5][i]); h[i][3] = pk(v[6][i], v[7][i]); }
        o += (sub >> 1) * 4096 + (sub & 1) * 1024; s1 = 2048; s2 = 16384;
    }
#pragma unroll 1
    for (int q = 0; q < NR; ++q) {
        const int d = (q + rank + sub) & 7;
        if (!((dmask >> d) & 1)) continue;
        const rsrc_t r = mk(p.ws[d] + bufoff, BUF_BYTES);
        __builtin_amdgcn_raw_buffer_store_b128(h[0], r, o, 0, AUX_SYS);
        __builtin_amdgcn_raw_buffer_store_b128(h[1], r, o + s1, 0, AUX_SYS);
        __builtin_amdgcn_raw_buffer_store_b128(h[2], r, o + s2, 0, AUX_SYS);
        __builtin_amdgcn_raw_buffer_store_b128(h[3], r, o + s1 + s2, 0, AUX_SYS);
    }
}
__device__ __forceinline__ int blk_at(const Params& p, int q) { return (p.blk[q >> 1] >> ((q & 1) * 16)) & 0xffff; }

struct Tile { int pm, pn; };
__device__ __forceinline__ bool slot(int j, int g, int x, int rank, Tile& t) {
    const int e = RJ * j + ((x & 1) ? GE : 0) + g, col = e >> 2, extra = (x >> 1) == (rank & 3);
    if (col > NPN + extra) return false;
    t.pm = 4 * (x >> 1) + (e & 3);
    t.pn = col < SPLIT_COL ? col : col == SPLIT_COL ? 120 + rank : extra && col == SPLIT_COL + 1 ? NPN + (rank >> 2) : col - 1 - extra;
    return true;
}

#define SA(b, h) (((b) * 2 + (h)) * HTB)
#define SB(b, h) ((4 + (b) * 2 + (h)) * HTB)
#define STAGE(bufoff, soff) do { \
    __builtin_amdgcn_raw_ptr_buffer_load_lds(rAB, (LAS void*)(lds + (bufoff) + ldsw), 16, voff, (soff), 0, AUX_SYS); \
    __builtin_amdgcn_raw_ptr_buffer_load_lds(rAB, (LAS void*)(lds + (bufoff) + ldsw), 16, voff, (soff), 1024, AUX_SYS); } while (0)
#define LDA(dst, b, h) do { _Pragma("unroll") for (int m = 0; m < 4; ++m) _Pragma("unroll") for (int k = 0; k < 2; ++k) dst[m][k] = *(const LAS h8*)(lds + SA(b, h) + aoff + m * 2048 + k * 1024); } while (0)
#define LDB(dst, b, h) do { _Pragma("unroll") for (int n = 0; n < 2; ++n) _Pragma("unroll") for (int k = 0; k < 2; ++k) dst[n][k] = *(const LAS h8*)(lds + SB(b, h) + boff + n * 2048 + k * 1024); } while (0)
#define MMA(ai, bj, At, Bt) do { __builtin_amdgcn_s_setprio(1); _Pragma("unroll") for (int m = 0; m < 4; ++m) _Pragma("unroll") for (int n = 0; n < 2; ++n) _Pragma("unroll") for (int k = 0; k < 2; ++k) { \
    acc[ai][bj][m][n] = __builtin_amdgcn_mfma_f32_16x16x32_f16(At[m][k], Bt[n][k], acc[ai][bj][m][n], 0, 0, 0); SCHED; } __builtin_amdgcn_s_setprio(0); } while (0)

__global__ __launch_bounds__(512, 2) void fused_gemm(Params p) {
    extern __shared__ __attribute__((aligned(16))) unsigned char shm[];
    LAS unsigned char* lds = (LAS unsigned char*)shm;
    const int tid = threadIdx.x, wid = __builtin_amdgcn_readfirstlane(tid >> 6), lane = tid & 63;
    const int x = blockIdx.x & 7, c = blockIdx.x >> 3, rank = p.rank;
    unsigned* ctrl = (unsigned*)p.ws[rank];
    if (wid == 0) { STAMP(64); STAMPV(66, __builtin_amdgcn_s_memtime()); }
    const unsigned long long tstart = __builtin_amdgcn_s_memrealtime();
    const unsigned done = __hip_atomic_load(ctrl + W_DONE, __ATOMIC_RELAXED, __HIP_MEMORY_SCOPE_AGENT);
    const int g = c - ((x & 1) ? PUSH_O : PUSH_E);
    const int q0 = g < 0 ? (x >> 1) * (PUSH_E + PUSH_O) + ((x & 1) ? PUSH_E : 0) + c : NPUSH + (g < GO ? g * 8 + x : GO * 8 + (g - GO) * 4 + (x >> 1));
    int b0 = blk_at(p, q0);
    f32x4 cv[8];
    conv_load(p, b0 >> 3, b0 & 7, wid, lane, cv);
    const unsigned call = __builtin_amdgcn_readfirstlane(done) / GRID + 1;
    const unsigned par = call & 1, ncp = (call + 1) >> 1;
    const size_t bufoff = CTRL_BYTES + (size_t)par * BUF_BYTES;
    const int pw = W_PAR + par * PAR_WORDS;
    conv_store(p, b0 >> 3, b0 & 7, wid, rank, bufoff, g < 0 ? 0xff : 1 << rank, lane, cv);
    if (blockIdx.x == 0 && tid == 0) __hip_atomic_store(ctrl + W_PAR + (par ^ 1) * PAR_WORDS + P_TICKET, 0u, __ATOMIC_RELAXED, __HIP_MEMORY_SCOPE_AGENT);

    if (g >= 0) {
        WAIT_V(0); BAR;
        if (tid < 2) __hip_atomic_fetch_add(ctrl + pw + (lane ? P_CNT + (b0 >> 3) * 32 + (b0 & 7) : q0 < NPUSH + 32 ? P_PRE : P_PRE + 32), lane ? 8u : 1u, __ATOMIC_RELAXED, __HIP_MEMORY_SCOPE_SYSTEM);
        if (wid == 0) STAMP(110);
        int word = P_PRE + 64, pass = 1;
        for (int q = q0 + PRE_WGS; q < NBLK; q += PRE_WGS, ++pass) {
            const int b = blk_at(p, q);
            while (__builtin_amdgcn_s_memrealtime() - tstart < (unsigned)(pass * CONV_GAP + (g & 7) * (CONV_GAP / 8))) __builtin_amdgcn_s_sleep(8);
            conv_load(p, b >> 3, b & 7, wid, lane, cv);
            conv_store(p, b >> 3, b & 7, wid, rank, bufoff, 1 << rank, lane, cv);
            word = lane == pass ? P_CNT + (b >> 3) * 32 + (b & 7) : word;
        }
        WAIT_V(0); BAR;
        if (tid < pass) __hip_atomic_fetch_add(ctrl + pw + word, lane ? 8u : 1u, __ATOMIC_RELAXED, __HIP_MEMORY_SCOPE_SYSTEM);
        STAMP(100 + wid);
    } else {
        unsigned char* sigb = p.ws[0];
#pragma unroll
        for (int d = 1; d < NR; ++d) sigb = lane == d ? p.ws[d] : sigb;
        unsigned* const sig = (unsigned*)sigb + pw;
        WAIT_V(0);
        if (lane < NR) __hip_atomic_fetch_add(sig + P_CNT + (b0 >> 3) * 32 + (b0 & 7), 1u, __ATOMIC_RELAXED, __HIP_MEMORY_SCOPE_SYSTEM);
        if (wid < PUSH_WAVES) {
            unsigned* const ticket = ctrl + pw + P_TICKET;
            const bool sigl = lane < NR && lane != rank;
            const int rot = wid + q0 * PUSH_WAVES;
            rsrc_t rd[7], rs = mk(p.ws[rank] + bufoff, BUF_BYTES);
#pragma unroll
            for (int q = 0; q < 7; ++q) rd[q] = mk(p.ws[(rank + 1 + (q + rot) % 7) & 7] + bufoff, BUF_BYTES);
            u32x4 v[4] = {}, h[4];
            unsigned tk = 0;
#define DRAW(n) do { if (lane == 0) asm volatile("global_atomic_add %0, %1, %2, off sc0" : "+v"(tk) : "v"(ticket), "v"(n) : "memory"); } while (0)
            DRAW(1u);
            int stage = 0;
            int word = 0, pword = -1; unsigned off = 0, ps = 1024;
            bool have = false, cur_fwd = false;
            WAIT_V(0);
            for (;;) {
                asm volatile("s_waitcnt vmcnt(29)" : "+v"(tk) :: "memory");
#pragma unroll
                for (int i = 0; i < 4; ++i) h[i] = v[i];
                const int u = __builtin_amdgcn_readfirstlane(tk);
                const bool valid = u < U_END, fwd = valid && u >= U_FWD;
                const int st = !valid ? 0 : fwd ? 4 : u >= PRE_WGS * 8 ? 3 : u >= 32 * 8 ? 2 : 1;
                if (st > stage) {
                    wait_mask(ctrl + pw + (st == 4 ? P_TILE : P_PRE + (st - 1) * 32), (st == 4 ? TILE_PER_CALL : st == 3 ? PRE_WGS : st == 2 ? PRE_WGS - 32 : 32) * ncp, 1, ctrl + W_TMO, call);
                    stage = st; if (wid == 0) STAMP(110 + st);
                    if (st == 4) rs = mk(p.out[rank], OUT_BYTES);
                }
                const int b = blk_at(p, valid && !fwd ? NPUSH + (u >> 3) : 0), fv = u - U_FWD;
                const int nword = fwd ? P_FWD + rank * 32 : P_CNT + (b >> 3) * 32 + (b & 7);
                const unsigned noff = !fwd ? (b >> 3) * PANEL + ((b & 7) * 8 + rank) * KTB + (u & 7) * 4096
                                    : fv < 1024 ? (unsigned)fv * 4u * (N * 4) + (120 + rank) * 1024
                                                : (unsigned)((rank & 3) * 256 + fv - 1024) * 4u * (N * 4) + (NPN + (rank >> 2)) * 1024;
                const unsigned nps = fwd ? N * 4 : 1024;
                DRAW(valid ? 1u : 0u);
#pragma unroll
                for (int i = 0; i < 4; ++i) v[i] = __builtin_amdgcn_raw_buffer_load_b128(rs, noff + lane * 16 + i * nps, 0, AUX_SYS);
                const unsigned o = (have ? off : 0x80000000u) + lane * 16;
#pragma unroll
                for (int i = 0; i < 4; ++i) {
#pragma unroll
                    for (int q = 0; q < 7; ++q) __builtin_amdgcn_raw_buffer_store_b128(h[i], rd[q], o + i * ps, 0, AUX_SYS);
                    if (i == 1) {
                        WAIT_V(19);
                        if (sigl) __hip_atomic_fetch_add(sig + (pword < 0 ? P_PRE + 96 : pword), 1u, __ATOMIC_RELAXED, __HIP_MEMORY_SCOPE_SYSTEM);
                    }
                }
                pword = have ? word : -1;
                if (!valid) break;
                if (fwd && !cur_fwd) {
#pragma unroll
                    for (int q = 0; q < 7; ++q) rd[q] = mk(p.out[(rank + 1 + (q + rot) % 7) & 7], OUT_BYTES);
                    cur_fwd = true;
                }
                word = nword; off = noff; ps = nps; have = true;
            }
            asm volatile("s_waitcnt vmcnt(0)" : "+v"(tk) :: "memory");
            if (sigl && pword >= 0) __hip_atomic_fetch_add(sig + pword, 1u, __ATOMIC_RELAXED, __HIP_MEMORY_SCOPE_SYSTEM);
            STAMP(100 + wid);
        }
        if (wid == 2) {
            wait_mask(ctrl + pw + P_TICKET, p.nbs * 8, 1, ctrl + W_TMO, call, 1);
            for (int q = q0; q < 80; q += NPUSH) {
                const int it = q < 64 ? 136 + ((q + rank) & 7) : 16 + NPN + (q & 1), sl = q < 64 ? q >> 3 : (q - 64) >> 1;
                const int dmask = q < 64 ? 1 << (it - 136) : 0xf << (4 * (q & 1));
                for (int sub = 0; sub < 8; ++sub) { conv_load(p, it, sl, sub, lane, cv); conv_store(p, it, sl, sub, rank, bufoff, dmask, lane, cv); }
                WAIT_V(0);
                if (lane < NR && ((dmask >> lane) & 1)) __hip_atomic_fetch_add(sig + P_CNT + it * 32 + sl, 8u, __ATOMIC_RELAXED, __HIP_MEMORY_SCOPE_SYSTEM);
            }
            STAMP(102);
        }
    }
    if (blockIdx.x == 1 && wid == 3) {
        if (!wait_mask(ctrl + pw + P_FWD + (lane & 7) * 32, FWD_PER_CALL * ncp, 0xff & ~(1 << rank), ctrl + W_TMO, call, 64) && lane == 0) p.out[rank][0] = __builtin_nanf("");
        __builtin_amdgcn_fence(__ATOMIC_ACQUIRE, "");
    }
    if (g >= 0) {
        const int wr = wid >> 2, wc = wid & 3, fr = lane & 15, fq = lane >> 4;
        const rsrc_t rAB = mk(p.ws[rank] + bufoff, BUF_BYTES), rCt = mk(ctrl, (unsigned)CTRL_BYTES), rC = mk(p.out[rank], OUT_BYTES);
        const unsigned voff = wid * 2048 + lane * 16, ldsw = wid * 2048;
        const int aoff = wr * 8192 + lane * 16, boff = wc * 4096 + lane * 16;
        const unsigned cvoff = (unsigned)(wr * 64 + 4 * fq) * (N * 4) + (wc * 64 + 4 * fr) * 4;
        const unsigned thr = CNT_PER_CALL * ncp;
        int j = 0; Tile cur, nxt;
        slot(j, g, x, rank, cur);
        bool bad = false;
        unsigned have = 0;
        if (wid == 0) {
            const int ln = (lane & 8) ? cur.pm : 16 + cur.pn;
            have = (unsigned)wait_mask(ctrl + pw + P_CNT + ln * 32 + (lane & 7), thr, 0x101, ctrl + W_TMO, call, 1) & 0xffffu;
            bad = have == 0;
            STAMP(68);
        }
        asm volatile("" ::: "memory"); BAR; asm volatile("" ::: "memory");
        unsigned long long wsum = 0; (void)wsum;
        f32x4 acc[2][2][4][2];
#pragma unroll
        for (int a = 0; a < 2; ++a)
#pragma unroll
            for (int b = 0; b < 2; ++b)
#pragma unroll
                for (int m = 0; m < 4; ++m)
#pragma unroll
                    for (int n = 0; n < 2; ++n) acc[a][b][m][n] = (f32x4){0.f, 0.f, 0.f, 0.f};
        h8 At[4][2], B0[2][2], B1[2][2];
        unsigned cA = cur.pm * PANEL, cB = A_BYTES + cur.pn * PANEL;
        STAGE(SB(0, 0), cB); STAGE(SB(0, 1), cB + HTB); STAGE(SA(0, 0), cA); STAGE(SA(0, 1), cA + HTB);
        if (wr == 1) BAR;
        WAIT_V(2); BAR;
        STAGE(SB(1, 0), cB + KTB); STAGE(SA(1, 0), cA + KTB); STAGE(SB(1, 1), cB + HTB + KTB);
        WAIT_V(6); BAR;
        for (;;) {
            const bool has_next = slot(j + 1, g, x, rank, nxt);
            if (!has_next) nxt = cur;
            const unsigned nA = nxt.pm * PANEL, nB = A_BYTES + nxt.pn * PANEL;
            const int sel = (lane >> 3) & 3, ln = sel == 0 ? 16 + cur.pn : sel == 1 ? cur.pm : sel == 2 ? 16 + nxt.pn : nxt.pm;
            const unsigned fw = pw + P_CNT + ln * 32 + (lane & 7);
            for (int t = 0; t < NKT; t += 2) {
                const bool last = (t == NKT - 2);
                const unsigned a1 = cA + (t + 1) * KTB;
                const unsigned a2 = last ? nA : cA + (t + 2) * KTB, b2 = last ? nB : cB + (t + 2) * KTB;
                const unsigned a3 = a2 + KTB, b3 = b2 + KTB;
                if (wid == 0 && (t & 4)) {
                    const unsigned need = t >= NKT - 4 ? 0x01010000u : 0x202u << (t >> 3);
                    if ((have & need) != need) {
                        if (!(t & 2)) __builtin_amdgcn_raw_ptr_buffer_load_lds(rCt, (LAS void*)(lds + 8 * HTB), 4, fw * 4, 0, 0, AUX_SYS);
                        else {
                            have = (unsigned)__ballot(*(volatile LAS unsigned*)(lds + 8 * HTB + lane * 4) >= thr);
                            if ((have & need) != need) {
                                have = (unsigned)wait_mask(ctrl + fw, thr, need, ctrl + W_TMO, call, 1);
                                bad |= have == 0;
                            }
                            asm volatile("" ::: "memory");
                        }
                    }
                }
                LDB(B0, 0, 0); LDB(B1, 0, 1); SCHED; LDA(At, 0, 0); STAGE(SA(1, 1), a1 + HTB);
                WAIT_V(8); WAIT_L(0); BAR; MMA(0, 0, At, B0); MMA(0, 1, At, B1); BAR; SCHED;
                LDA(At, 0, 1); STAGE(SB(0, 0), b2); STAGE(SB(0, 1), b2 + HTB); STAGE(SA(0, 0), a2);
                WAIT_V(8); WAIT_L(0); BAR; MMA(1, 0, At, B0); MMA(1, 1, At, B1); BAR; SCHED;
                LDB(B0, 1, 0); LDB(B1, 1, 1); SCHED; LDA(At, 1, 0); STAGE(SA(0, 1), a2 + HTB);
                WAIT_V(8); WAIT_L(0); BAR; MMA(0, 0, At, B0); MMA(0, 1, At, B1); BAR; SCHED;
                LDA(At, 1, 1); STAGE(SB(1, 0), b3); STAGE(SB(1, 1), b3 + HTB); STAGE(SA(1, 0), a3);
                WAIT_V(8); WAIT_L(0); BAR; MMA(1, 0, At, B0); MMA(1, 1, At, B1); BAR; SCHED;
            }
            if (wid == 0) { STAMP(72 + j); STAMPV(88 + j, wsum); wsum = 0; }
            if (wr == 0) BAR;
            if (bad) acc[0][0][0][0][0] = __builtin_nanf("");
            const unsigned cb = cvoff + (unsigned)cur.pm * 256u * (N * 4) + (unsigned)cur.pn * 1024u;
#define C_AUX 0
            const bool own = cur.pn >= NPN;
#pragma unroll
            for (int ai = 0; ai < 2; ++ai)
#pragma unroll
                for (int m = 0; m < 4; ++m)
#pragma unroll
                    for (int q = 0; q < 4; ++q) {
                        const f32x4 o = {acc[ai][0][m][0][q], acc[ai][0][m][1][q], acc[ai][1][m][0][q], acc[ai][1][m][1][q]};
                        const unsigned co = cb + (unsigned)(ai * 128 + m * 16 + q) * (N * 4);
                        if (C_AUX == AUX_SYS || own) __builtin_amdgcn_raw_buffer_store_b128(__builtin_bit_cast(u32x4, o), rC, co, 0, AUX_SYS);
                        else __builtin_amdgcn_raw_buffer_store_b128(__builtin_bit_cast(u32x4, o), rC, co, 0, C_AUX);
                    }
            if (own) {
                WAIT_V(0);
                if (lane == 0) __hip_atomic_fetch_add(ctrl + pw + P_TILE, 1u, __ATOMIC_RELAXED, __HIP_MEMORY_SCOPE_AGENT);
            }
            if (wid == 0) STAMP(80 + j);
            if (!has_next) break;
#pragma unroll
            for (int a = 0; a < 2; ++a)
#pragma unroll
                for (int b = 0; b < 2; ++b)
#pragma unroll
                    for (int m = 0; m < 4; ++m)
#pragma unroll
                        for (int n = 0; n < 2; ++n) acc[a][b][m][n] = (f32x4){0.f, 0.f, 0.f, 0.f};
            cur = nxt; cA = nA; cB = nB; ++j; have >>= 16;
            if (wr == 1) BAR;
        }
        WAIT_V(0);
    }
    BAR;
    if (wid == 0) { STAMP(65); STAMPV(67, __builtin_amdgcn_s_memtime()); }
    if (tid == 0) __hip_atomic_fetch_add(ctrl + W_DONE, 1u, __ATOMIC_RELAXED, __HIP_MEMORY_SCOPE_AGENT);
}

void make_order(Params& p) {
    int u = 0;
    for (int j = 0; j < 8; ++j) {
        int n = 0, cols[16]; double key[16];
        for (int col = j * 15; col < j * 15 + 15 && col <= NPN; ++col) {
            if (col == SPLIT_COL) continue;
            const int e = (col - j * 15) * 4;
            double k = e < GE ? (e + 0.5) / GE + ODD_FIRST : (double)(e - GE) / GO;
            if (e < GE && e + 3 >= GE) k = 0.0;
            int i = n++;
            while (i > 0 && key[i - 1] > k) { key[i] = key[i - 1]; cols[i] = cols[i - 1]; --i; }
            key[i] = k; cols[i] = col;
        }
        for (int sl = 0; sl < 8; ++sl) {
            if (j == 0) for (int a = 0; a < 16; ++a, ++u) p.blk[u >> 1] |= (unsigned)(a * 8 + sl) << ((u & 1) * 16);
            for (int i = 0; i < n; ++i, ++u) p.blk[u >> 1] |= (unsigned)((16 + cols[i] - (cols[i] > SPLIT_COL)) * 8 + sl) << ((u & 1) * 16);
        }
        if (j == 2) p.nbs = u - NPUSH;
    }
}
}

extern "C" void kernel_launch(int rank, int nranks, ncclComm_t comm, hipStream_t stream,
                              void* const* d_in, const int* in_sizes, int n_in,
                              void* d_out, int out_size, const int* in_dtypes, int out_dtype,
                              void* const* d_out_all, void* const* d_ws_all, size_t ws_bytes) {
    if (nranks != NR || (unsigned)rank >= (unsigned)NR || n_in != 2 || in_sizes[0] != M * KL || in_sizes[1] != KL * N
        || out_size != M * N || ws_bytes < WS_NEED) { fprintf(stderr, "unexpected problem shape\n"); return; }
    static bool init = false;
    if (!init) { HIP_CHECK(hipFuncSetAttribute((const void*)fused_gemm, hipFuncAttributeMaxDynamicSharedMemorySize, LDS_BYTES)); init = true; }
    Params p = {};
    p.x = (const float*)d_in[0]; p.w = (const float*)d_in[1]; p.rank = rank;
    for (int r = 0; r < NR; ++r) { p.out[r] = (float*)d_out_all[r]; p.ws[r] = (unsigned char*)d_ws_all[r]; }
    make_order(p);
    hipLaunchKernelGGL(fused_gemm, dim3(GRID), dim3(512), LDS_BYTES, stream, p);
    HIP_CHECK(hipGetLastError());
}
```

```cpp
#include <hip/hip_runtime.h>
#include <rccl/rccl.h>
#include <cstdio>

#define HIP_CHECK(expr) do { hipError_t _e = (expr); if (_e != hipSuccess) { \
    fprintf(stderr, "HIP error %s at %s:%d\n", hipGetErrorString(_e), __FILE__, __LINE__); \
    return; } } while (0)

namespace {
#define LAS __attribute__((address_space(3)))
typedef _Float16 h8 __attribute__((ext_vector_type(8)));
typedef _Float16 h2 __attribute__((ext_vector_type(2)));
typedef float f32x4 __attribute__((ext_vector_type(4)));
typedef unsigned u32x4 __attribute__((ext_vector_type(4)));
typedef __amdgpu_buffer_rsrc_t rsrc_t;

constexpr int NR = 8, M = 4096, N = 32768, KL = 512, NKT = 64, GRID = 256;
constexpr unsigned HTB = 16384, KTB = 32768, PANEL = NKT * KTB;
constexpr unsigned A_BYTES = (M / 256) * PANEL, B_BYTES = (N / 256) * PANEL, BUF_BYTES = A_BYTES + B_BYTES;
constexpr unsigned OUT_BYTES = 512u << 20, X_BYTES = M * KL * 4, W_BYTES = KL * N * 4;
constexpr size_t CTRL_BYTES = 1u << 20, WS_NEED = CTRL_BYTES + 2 * (size_t)BUF_BYTES;
constexpr int W_DONE = 0, W_TMO = 32, W_PAR = 64, PAR_WORDS = 160 * 32;
constexpr int P_TICKET = 0, P_TILE = 32, P_FWD = 64, P_PRE = 10 * 32, P_SPC = 13 * 32, P_CNT = 16 * 32;
constexpr unsigned CNT_PER_CALL = 64, TILE_PER_CALL = 256, FWD_PER_CALL = 2048;
constexpr unsigned long long DEADLINE = 50000000ull;
constexpr int LDS_BYTES = 8 * HTB + 1024;
constexpr int AUX_SYS = 17, AUX_NT = 2;
#define PUSH_E 2
#define PUSH_O 2
#define POLL_SLEEP 64
#define CONV_GAP 800
#define ODD_FIRST 0.0
#define FWD_MIX 3
#define SPEC_GAP 80
#define NDIR 12
#define K1_BLK 192
#define UPT 2
#define PUSH_WAVES 2
#define R0E GE
constexpr int GE = 32 - PUSH_E, GO = 32 - PUSH_O, RJ = GE + GO, NPN = 112, R0 = (NPN + 2) * 4 - 7 * RJ, R0O = R0 - R0E, SPLIT_COL = 0;
constexpr int PRE_WGS = 4 * RJ, NPUSH = 4 * (PUSH_E + PUSH_O);
constexpr int NBLK = (16 + NPN) * 8 + FWD_PER_CALL / 8, U_END = NBLK * 8, U_K1 = K1_BLK * 8;
static_assert(K1_BLK >= 8 * (14 + R0 / 4), "the paced wave counts tickets as units");
static_assert(GE >= GO && RJ == 60 && R0 % 4 == 0 && R0E <= GE && R0O >= 0 && R0O <= GO, "role split");

struct Params { const float* x; const float* w; float* out[NR]; unsigned char* ws[NR]; int rank; unsigned blk[NBLK / 2]; };

__device__ __forceinline__ rsrc_t mk(const void* p, unsigned bytes) { return __builtin_amdgcn_make_buffer_rsrc((void*)p, 0, bytes, 0x00020000); }
__device__ __forceinline__ unsigned pk(float a, float b) { h2 h; h.x = (_Float16)a; h.y = (_Float16)b; return __builtin_bit_cast(unsigned, h); }
#define WAIT_V(n) asm volatile("s_waitcnt vmcnt(" #n ")" ::: "memory")
#define WAIT_L(n) asm volatile("s_waitcnt lgkmcnt(" #n ")" ::: "memory")
#define BAR __builtin_amdgcn_s_barrier()
#define SCHED __builtin_amdgcn_sched_barrier(0)
#define STAMP(i) do {} while (0)
#define STAMPV(i, v) do {} while (0)

__device__ __forceinline__ unsigned long long wait_mask(unsigned* word, unsigned thr, unsigned long long need, unsigned* tmo, unsigned call, int slow = 0) {
    const unsigned long long t0 = __builtin_amdgcn_s_memrealtime();
    for (unsigned it = 1;; ++it) {
        const unsigned long long got = __ballot(__hip_atomic_load(word, __ATOMIC_RELAXED, __HIP_MEMORY_SCOPE_SYSTEM) >= thr);
        if ((got & need) == need) return got;
        if ((it & 63) == 0 && (__builtin_amdgcn_readfirstlane(__hip_atomic_load(tmo, __ATOMIC_RELAXED, __HIP_MEMORY_SCOPE_AGENT)) == call
            || __builtin_amdgcn_s_memrealtime() - t0 > DEADLINE)) {
            __hip_atomic_store(tmo, call, __ATOMIC_RELAXED, __HIP_MEMORY_SCOPE_AGENT);
            return 0;
        }
        if (slow) __builtin_amdgcn_s_sleep(POLL_SLEEP); else __builtin_amdgcn_s_sleep(4);
    }
}

__device__ __forceinline__ void conv_load(const Params& p, int it, int s, int sub, int lane, f32x4 (&v)[8]) {
    if (it < 16) {
        const rsrc_t r = mk(p.x, X_BYTES);
        const unsigned a = (unsigned)(it * 256 + sub * 32 + (lane & 15)) * (KL * 4) + s * 256 + (lane >> 4) * 32;
#pragma unroll
        for (int i = 0; i < 8; ++i) v[i] = __builtin_bit_cast(f32x4, __builtin_amdgcn_raw_buffer_load_b128(r, a + (i >> 2) * 16 * (KL * 4) + ((i >> 1) & 1) * 128 + (i & 1) * 16, 0, AUX_NT));
    } else {
        const rsrc_t r = mk(p.w, W_BYTES);
        const unsigned a = (unsigned)(s * 64 + (sub & 1) * 32 + (lane >> 4) * 8) * (N * 4) + (it - 16) * 1024 + (sub >> 1) * 256 + (lane & 15) * 16;
#pragma unroll
        for (int j = 0; j < 8; ++j) v[j] = __builtin_bit_cast(f32x4, __builtin_amdgcn_raw_buffer_load_b128(r, a + j * (N * 4), 0, AUX_NT));
    }
}
__device__ __forceinline__ void conv_store(const Params& p, int it, int s, int sub, int rank, size_t bufoff, int dmask, int rot, int lane, const f32x4 (&v)[8]) {
    u32x4 h[4];
    unsigned o = it * PANEL + (s * 8 + rank) * KTB + lane * 16, s1, s2;
    if (it < 16) {
#pragma unroll
        for (int i = 0; i < 4; ++i) { h[i][0] = pk(v[2 * i][0], v[2 * i][1]); h[i][1] = pk(v[2 * i][2], v[2 * i][3]); h[i][2] = pk(v[2 * i + 1][0], v[2 * i + 1][1]); h[i][3] = pk(v[2 * i + 1][2], v[2 * i + 1][3]); }
        o += sub * 4096; s1 = 1024; s2 = 2048;
    } else {
#pragma unroll
        for (int i = 0; i < 4; ++i) { h[i][0] = pk(v[0][i], v[1][i]); h[i][1] = pk(v[2][i], v[3][i]); h[i][2] = pk(v[4][i], v[5][i]); h[i][3] = pk(v[6][i], v[7][i]); }
        o += (sub >> 1) * 4096 + (sub & 1) * 1024; s1 = 2048; s2 = 16384;
    }
#pragma unroll 1
    for (int q = 0; q < NR; ++q) {
        const int d = (q + rot) & 7;
        if (!((dmask >> d) & 1)) continue;
        const rsrc_t r = mk(p.ws[d] + bufoff, BUF_BYTES);
        __builtin_amdgcn_raw_buffer_store_b128(h[0], r, o, 0, AUX_SYS);
        __builtin_amdgcn_raw_buffer_store_b128(h[1], r, o + s1, 0, AUX_SYS);
        __builtin_amdgcn_raw_buffer_store_b128(h[2], r, o + s2, 0, AUX_SYS);
        __builtin_amdgcn_raw_buffer_store_b128(h[3], r, o + s1 + s2, 0, AUX_SYS);
    }
}
__device__ __forceinline__ int blk_at(const Params& p, int q) { return (p.blk[q >> 1] >> ((q & 1) * 16)) & 0xffff; }

struct Tile { int pm, pn; };
__device__ __forceinline__ bool slot(int j, int g, int x, int rank, Tile& t) {
    if (j >= 8) return false;
    const int e = j == 0 ? ((x & 1) ? R0E : 0) + g : R0 + RJ * (j - 1) + ((x & 1) ? GE : 0) + g, col = e >> 2;
    t.pm = 4 * (x >> 1) + (e & 3);
    t.pn = col < SPLIT_COL ? col : col < SPLIT_COL + 2 ? NPN + (col - SPLIT_COL) * 8 + rank : col - 2;
    return true;
}

#define SA(b, h) (((b) * 2 + (h)) * HTB)
#define SB(b, h) ((4 + (b) * 2 + (h)) * HTB)
#define STAGE(bufoff, soff) do { \
    __builtin_amdgcn_raw_ptr_buffer_load_lds(rAB, (LAS void*)(lds + (bufoff) + ldsw), 16, voff, (soff), 0, AUX_SYS); \
    __builtin_amdgcn_raw_ptr_buffer_load_lds(rAB, (LAS void*)(lds + (bufoff) + ldsw), 16, voff, (soff), 1024, AUX_SYS); } while (0)
#define LDA(dst, b, h) do { _Pragma("unroll") for (int m = 0; m < 4; ++m) _Pragma("unroll") for (int k = 0; k < 2; ++k) dst[m][k] = *(const LAS h8*)(lds + SA(b, h) + aoff + m * 2048 + k * 1024); } while (0)
#define LDB(dst, b, h) do { _Pragma("unroll") for (int n = 0; n < 2; ++n) _Pragma("unroll") for (int k = 0; k < 2; ++k) dst[n][k] = *(const LAS h8*)(lds + SB(b, h) + boff + n * 2048 + k * 1024); } while (0)
#define MMA(ai, bj, At, Bt) do { __builtin_amdgcn_s_setprio(1); _Pragma("unroll") for (int m = 0; m < 4; ++m) _Pragma("unroll") for (int n = 0; n < 2; ++n) _Pragma("unroll") for (int k = 0; k < 2; ++k) { \
    acc[ai][bj][m][n] = __builtin_amdgcn_mfma_f32_16x16x32_f16(At[m][k], Bt[n][k], acc[ai][bj][m][n], 0, 0, 0); SCHED; } __builtin_amdgcn_s_setprio(0); } while (0)

__global__ __launch_bounds__(512, 2) void fused_gemm(Params p) {
    extern __shared__ __attribute__((aligned(16))) unsigned char shm[];
    LAS unsigned char* lds = (LAS unsigned char*)shm;
    const int tid = threadIdx.x, wid = __builtin_amdgcn_readfirstlane(tid >> 6), lane = tid & 63;
    const int x = blockIdx.x & 7, c = blockIdx.x >> 3, rank = p.rank;
    unsigned* ctrl = (unsigned*)p.ws[rank];
    if (wid == 0) { STAMP(64); STAMPV(66, __builtin_amdgcn_s_memtime()); }
    const unsigned long long tstart = __builtin_amdgcn_s_memrealtime();
    const unsigned done = __hip_atomic_load(ctrl + W_DONE, __ATOMIC_RELAXED, __HIP_MEMORY_SCOPE_AGENT);
    const int g = c - ((x & 1) ? PUSH_O : PUSH_E);
    const int q0 = g < 0 ? (x >> 1) * (PUSH_E + PUSH_O) + ((x & 1) ? PUSH_E : 0) + c : g < GO ? g * 8 + x : GO * 8 + (g - GO) * 4 + (x >> 1);
    const int b0 = blk_at(p, q0);
    f32x4 cv[8];
    if (g >= 0) conv_load(p, b0 >> 3, b0 & 7, wid, lane, cv);
    const unsigned call = __builtin_amdgcn_readfirstlane(done) / GRID + 1;
    const unsigned par = call & 1, ncp = (call + 1) >> 1;
    const size_t bufoff = CTRL_BYTES + (size_t)par * BUF_BYTES;
    const int pw = W_PAR + par * PAR_WORDS;
    if (blockIdx.x == 0 && tid == 0) __hip_atomic_store(ctrl + W_PAR + (par ^ 1) * PAR_WORDS + P_TICKET, 0u, __ATOMIC_RELAXED, __HIP_MEMORY_SCOPE_AGENT);
    unsigned char* sigb = p.ws[0];
#pragma unroll
    for (int d = 1; d < NR; ++d) sigb = lane == d ? p.ws[d] : sigb;
    unsigned* const sig = (unsigned*)sigb + pw;

    if (g >= 0) {
        conv_store(p, b0 >> 3, b0 & 7, wid, rank, bufoff, q0 < NDIR ? 0xff : 1 << rank, rank + q0 * 3 + wid * 2 + (wid >> 2), lane, cv);
        WAIT_V(0); BAR;
        if (tid < NR && (q0 < NDIR || tid == rank)) __hip_atomic_fetch_add(sig + P_CNT + (b0 >> 3) * 32 + (b0 & 7), 8u, __ATOMIC_RELAXED, __HIP_MEMORY_SCOPE_SYSTEM);
        if (tid == 64 && q0 >= NDIR) __hip_atomic_fetch_add(ctrl + pw + (q0 < NDIR + 32 ? P_PRE : P_PRE + 32), 1u, __ATOMIC_RELAXED, __HIP_MEMORY_SCOPE_SYSTEM);
        if (wid == 0) STAMP(110);
        const int gi = g - ((x & 1) ? R0O : R0E);
        for (int q = gi < 0 ? 128 : (((x & 1) ? GE - R0E : 0) + gi) * 4 + (x >> 1); q < 128; q += 4 * (RJ - R0)) {
            const int it = 16 + NPN + (q & 15), sl = q >> 4;
            conv_load(p, it, sl, wid, lane, cv); conv_store(p, it, sl, wid, rank, bufoff, 1 << rank, rank, lane, cv);
            WAIT_V(0); BAR;
            if (tid == 0 || (tid == 1 && (q & 7) == rank)) __hip_atomic_fetch_add(ctrl + pw + (tid ? P_CNT + it * 32 + sl : P_SPC), tid ? 8u : 1u, __ATOMIC_RELAXED, __HIP_MEMORY_SCOPE_SYSTEM);
        }
        int word = P_PRE + 64, pass = 1;
        for (int q = q0 + PRE_WGS; q < NBLK; q += PRE_WGS) {
            const int b = blk_at(p, q);
            if (b & 0x8000) continue;
            while (__builtin_amdgcn_s_memrealtime() - tstart < (unsigned)(pass * CONV_GAP + (g & 7) * (CONV_GAP / 8))) __builtin_amdgcn_s_sleep(8);
            conv_load(p, b >> 3, b & 7, wid, lane, cv);
            conv_store(p, b >> 3, b & 7, wid, rank, bufoff, 1 << rank, rank, lane, cv);
            word = lane == pass ? P_CNT + (b >> 3) * 32 + (b & 7) : word;
            ++pass;
        }
        WAIT_V(0); BAR;
        if (tid < pass) __hip_atomic_fetch_add(ctrl + pw + word, lane ? 8u : 1u, __ATOMIC_RELAXED, __HIP_MEMORY_SCOPE_SYSTEM);
        STAMP(100 + wid);
    } else {
        if (wid < PUSH_WAVES) {
            unsigned* const ticket = ctrl + pw + P_TICKET;
            const bool sigl = lane < NR && lane != rank;
            const int rot = wid + q0 * PUSH_WAVES;
            unsigned char* pd[7]; unsigned char* po[7];
#pragma unroll
            for (int q = 0; q < 7; ++q) { const int d = (rank + 1 + (q + rot) % 7) & 7; pd[q] = p.ws[d] + bufoff; po[q] = (unsigned char*)p.out[d]; }
            u32x4 v[4] = {}, h[4];
            unsigned tk = 0;
#define SIGNAL(w) do { if ((w) >= 0 && sigl) __hip_atomic_fetch_add(sig + ((w) >> 4), (unsigned)((w) & 15), __ATOMIC_RELAXED, __HIP_MEMORY_SCOPE_SYSTEM); } while (0)
#define DRAW(n) do { if (lane == 0) asm volatile("global_atomic_add %0, %1, %2, off sc0" : "+v"(tk) : "v"(ticket), "v"((unsigned)(n)) : "memory"); } while (0)
            DRAW(1);
            int stage = 0;
            int u = 0, left = 0, word = -1, pword = -1, ppword = -1;
            unsigned off = 0x80000000u, ps = 1024;
            bool cur_fwd = false, tiles_ok = false;
            WAIT_V(0);
            for (;;) {
                asm volatile("s_waitcnt vmcnt(28)" : "+v"(tk) :: "memory");
                SIGNAL(ppword);
#pragma unroll
                for (int i = 0; i < 4; ++i) h[i] = v[i];
                const bool fresh = left == 0;
                if (fresh) { const int k = __builtin_amdgcn_readfirstlane(tk) + NDIR * 8; u = k < U_K1 ? k : U_K1 + (k - U_K1) * UPT; left = k < U_K1 ? 0 : UPT - 1; }
                else { ++u; --left; }
                const bool valid = u < U_END;
                const int b = blk_at(p, valid ? u >> 3 : 0), fv = (b & 0xff) * 8 + (u & 7);
                const bool fwd = b & 0x8000;
                const int st = !valid ? 0 : u >= PRE_WGS * 8 ? 3 : u >= (NDIR + 32) * 8 ? 2 : 1;
                if (st > stage) {
                    wait_mask(ctrl + pw + P_PRE + (st - 1) * 32, (st == 3 ? PRE_WGS : st == 2 ? PRE_WGS - 32 - NDIR : 32) * ncp, 1, ctrl + W_TMO, call);
                    stage = st; if (wid == 0) STAMP(110 + st);
                }
                if (fwd && !tiles_ok) { wait_mask(ctrl + pw + P_TILE, TILE_PER_CALL * ncp, 1, ctrl + W_TMO, call); tiles_ok = true; if (wid == 0) STAMP(114); }
                const int nword = left ? -1 : ((fwd ? P_FWD + rank * 32 : P_CNT + (b >> 3) * 32 + (b & 7)) << 4) + (u < U_K1 ? 1 : UPT);
                const unsigned noff = !fwd ? (b >> 3) * PANEL + ((b & 7) * 8 + rank) * KTB + (u & 7) * 4096
                                           : (unsigned)(fv & 1023) * 4u * (N * 4) + (NPN + (fv >> 10) * 8 + rank) * 1024;
                const unsigned nps = fwd ? N * 4 : 1024;
                if (fresh && valid) DRAW(1);
                const rsrc_t rs = mk(fwd ? (unsigned char*)p.out[rank] : p.ws[rank] + bufoff, fwd ? OUT_BYTES : BUF_BYTES);
#pragma unroll
                for (int i = 0; i < 4; ++i) v[i] = __builtin_amdgcn_raw_buffer_load_b128(rs, noff + lane * 16 + i * nps, 0, AUX_SYS);
                rsrc_t rd[7];
#pragma unroll
                for (int q = 0; q < 7; ++q) rd[q] = mk(cur_fwd ? po[q] : pd[q], cur_fwd ? OUT_BYTES : BUF_BYTES);
#pragma unroll
                for (int i = 0; i < 4; ++i)
#pragma unroll
                    for (int q = 0; q < 7; ++q) __builtin_amdgcn_raw_buffer_store_b128(h[i], rd[q], off + lane * 16 + i * ps, 0, AUX_SYS);
                ppword = pword; pword = word;
                if (!valid) break;
                word = nword; off = noff; ps = nps; cur_fwd = fwd;
            }
            asm volatile("s_waitcnt vmcnt(0)" : "+v"(tk) :: "memory");
            SIGNAL(ppword); SIGNAL(pword);
            STAMP(100 + wid);
        }
        if (wid == 2) {
            static_assert(NPUSH == 16, "one pusher per (panel set, unit)");
            const rsrc_t rl = mk(p.ws[rank] + bufoff, BUF_BYTES);
            const int it = 16 + NPN + (q0 & 8);
            wait_mask(ctrl + pw + P_SPC, 128 * ncp, 1, ctrl + W_TMO, call);
            for (int sl = 0; sl < 8; ++sl) {
                const unsigned off = it * PANEL + (sl * 8 + rank) * KTB + (q0 & 7) * 4096 + lane * 16;
                u32x4 z[7][4];
#pragma unroll
                for (int q = 0; q < 7; ++q)
#pragma unroll
                    for (int i = 0; i < 4; ++i) z[q][i] = __builtin_amdgcn_raw_buffer_load_b128(rl, off + ((rank + 1 + (q + q0) % 7) & 7) * PANEL + i * 1024, 0, AUX_SYS);
                wait_mask(ctrl + pw + P_TICKET, (sl * (14 + R0 / 4) + q0 - NDIR > 0 ? sl * (14 + R0 / 4) + q0 - NDIR : 0) * 8, 1, ctrl + W_TMO, call, 1);
#pragma unroll
                for (int i = 0; i < 4; ++i)
#pragma unroll
                    for (int q = 0; q < 7; ++q) { const int d = (rank + 1 + (q + q0) % 7) & 7; __builtin_amdgcn_raw_buffer_store_b128(z[q][i], mk(p.ws[d] + bufoff, BUF_BYTES), off + d * PANEL + i * 1024, 0, AUX_SYS); }
                WAIT_V(0);
                if (lane < NR && lane != rank) __hip_atomic_fetch_add(sig + P_CNT + (it + lane) * 32 + sl, 1u, __ATOMIC_RELAXED, __HIP_MEMORY_SCOPE_SYSTEM);
            }
            STAMP(102);
        }
    }
    if (blockIdx.x == 1 && wid == 5) {
        if (!wait_mask(ctrl + pw + P_FWD + (lane & 7) * 32, FWD_PER_CALL * ncp, 0xff & ~(1 << rank), ctrl + W_TMO, call, 64) && lane == 0) p.out[rank][0] = __builtin_nanf("");
        __builtin_amdgcn_fence(__ATOMIC_ACQUIRE, "");
    }
    if (g >= 0) {
        const int wr = wid >> 2, wc = wid & 3, fr = lane & 15, fq = lane >> 4;
        const rsrc_t rAB = mk(p.ws[rank] + bufoff, BUF_BYTES), rCt = mk(ctrl, (unsigned)CTRL_BYTES), rC = mk(p.out[rank], OUT_BYTES);
        const unsigned voff = wid * 2048 + lane * 16, ldsw = wid * 2048;
        const int aoff = wr * 8192 + lane * 16, boff = wc * 4096 + lane * 16;
        const unsigned cvoff = (unsigned)(wr * 64 + 4 * fq) * (N * 4) + (wc * 64 + 4 * fr) * 4;
        const unsigned thr = CNT_PER_CALL * ncp;
        int j = g < ((x & 1) ? R0O : R0E) ? 0 : 1; Tile cur, nxt;
        slot(j, g, x, rank, cur);
        bool bad = false;
        unsigned have = 0;
        if (wid == 0) {
            const int ln = (lane & 8) ? cur.pm : 16 + cur.pn;
            have = (unsigned)wait_mask(ctrl + pw + P_CNT + ln * 32 + (lane & 7), thr, 0x101, ctrl + W_TMO, call, 1) & 0xffffu;
            bad = have == 0;
            STAMP(68);
        }
        asm volatile("" ::: "memory"); BAR; asm volatile("" ::: "memory");
        unsigned long long wsum = 0; (void)wsum;
        f32x4 acc[2][2][4][2];
#pragma unroll
        for (int a = 0; a < 2; ++a)
#pragma unroll
            for (int b = 0; b < 2; ++b)
#pragma unroll
                for (int m = 0; m < 4; ++m)
#pragma unroll
                    for (int n = 0; n < 2; ++n) acc[a][b][m][n] = (f32x4){0.f, 0.f, 0.f, 0.f};
        h8 At[4][2], B0[2][2], B1[2][2];
        unsigned cA = cur.pm * PANEL, cB = A_BYTES + cur.pn * PANEL;
        STAGE(SB(0, 0), cB); STAGE(SB(0, 1), cB + HTB); STAGE(SA(0, 0), cA); STAGE(SA(0, 1), cA + HTB);
        if (wr == 1) BAR;
        WAIT_V(2); BAR;
        STAGE(SB(1, 0), cB + KTB); STAGE(SA(1, 0), cA + KTB); STAGE(SB(1, 1), cB + HTB + KTB);
        WAIT_V(6); BAR;
        for (;;) {
            const bool has_next = slot(j + 1, g, x, rank, nxt);
            if (!has_next) nxt = cur;
            const unsigned nA = nxt.pm * PANEL, nB = A_BYTES + nxt.pn * PANEL;
            const int sel = (lane >> 3) & 3, ln = sel == 0 ? 16 + cur.pn : sel == 1 ? cur.pm : sel == 2 ? 16 + nxt.pn : nxt.pm;
            const unsigned fw = pw + P_CNT + ln * 32 + (lane & 7);
            for (int t = 0; t < NKT; t += 2) {
                const bool last = (t == NKT - 2);
                const unsigned a1 = cA + (t + 1) * KTB;
                const unsigned a2 = last ? nA : cA + (t + 2) * KTB, b2 = last ? nB : cB + (t + 2) * KTB;
                const unsigned a3 = a2 + KTB, b3 = b2 + KTB;
                if (wid == 0 && (t & 4)) {
                    const unsigned need = t >= NKT - 4 ? 0x01010000u : 0x202u << (t >> 3);
                    if ((have & need) != need) {
                        if (!(t & 2)) __builtin_amdgcn_raw_ptr_buffer_load_lds(rCt, (LAS void*)(lds + 8 * HTB), 4, fw * 4, 0, 0, AUX_SYS);
                        else {
                            have = (unsigned)__ballot(*(volatile LAS unsigned*)(lds + 8 * HTB + lane * 4) >= thr);
                            if ((have & need) != need) {
                                have = (unsigned)wait_mask(ctrl + fw, thr, need, ctrl + W_TMO, call, 1);
                                bad |= have == 0;
                            }
                            asm volatile("" ::: "memory");
                        }
                    }
                }
                LDB(B0, 0, 0); LDB(B1, 0, 1); SCHED; LDA(At, 0, 0); STAGE(SA(1, 1), a1 + HTB);
                WAIT_V(8); WAIT_L(0); BAR; MMA(0, 0, At, B0); MMA(0, 1, At, B1); BAR; SCHED;
                LDA(At, 0, 1); STAGE(SB(0, 0), b2); STAGE(SB(0, 1), b2 + HTB); STAGE(SA(0, 0), a2);
                WAIT_V(8); WAIT_L(0); BAR; MMA(1, 0, At, B0); MMA(1, 1, At, B1); BAR; SCHED;
                LDB(B0, 1, 0); LDB(B1, 1, 1); SCHED; LDA(At, 1, 0); STAGE(SA(0, 1), a2 + HTB);
                WAIT_V(8); WAIT_L(0); BAR; MMA(0, 0, At, B0); MMA(0, 1, At, B1); BAR; SCHED;
                LDA(At, 1, 1); STAGE(SB(1, 0), b3); STAGE(SB(1, 1), b3 + HTB); STAGE(SA(1, 0), a3);
                WAIT_V(8); WAIT_L(0); BAR; MMA(1, 0, At, B0); MMA(1, 1, At, B1); BAR; SCHED;
            }
            if (wid == 0) { STAMP(72 + j); STAMPV(88 + j, wsum); wsum = 0; STAMPV(116 + j, __builtin_amdgcn_s_memtime()); }
            if (wr == 0) BAR;
            if (bad) acc[0][0][0][0][0] = __builtin_nanf("");
            const unsigned cb = cvoff + (unsigned)cur.pm * 256u * (N * 4) + (unsigned)cur.pn * 1024u;
#define C_AUX AUX_NT
            const bool own = cur.pn >= NPN;
#pragma unroll
            for (int ai = 0; ai < 2; ++ai)
#pragma unroll
                for (int m = 0; m < 4; ++m)
#pragma unroll
                    for (int q = 0; q < 4; ++q) {
                        const f32x4 o = {acc[ai][0][m][0][q], acc[ai][0][m][1][q], acc[ai][1][m][0][q], acc[ai][1][m][1][q]};
                        const unsigned co = cb + (unsigned)(ai * 128 + m * 16 + q) * (N * 4);
                        if (C_AUX == AUX_SYS || own) __builtin_amdgcn_raw_buffer_store_b128(__builtin_bit_cast(u32x4, o), rC, co, 0, AUX_SYS);
                        else __builtin_amdgcn_raw_buffer_store_b128(__builtin_bit_cast(u32x4, o), rC, co, 0, C_AUX);
                    }
            if (own) {
                WAIT_V(0);
                if (lane == 0) __hip_atomic_fetch_add(ctrl + pw + P_TILE, 1u, __ATOMIC_RELAXED, __HIP_MEMORY_SCOPE_AGENT);
            }
            if (wid == 0) STAMP(80 + j);
            if (!has_next) break;
#pragma unroll
            for (int a = 0; a < 2; ++a)
#pragma unroll
                for (int b = 0; b < 2; ++b)
#pragma unroll
                    for (int m = 0; m < 4; ++m)
#pragma unroll
                        for (int n = 0; n < 2; ++n) acc[a][b][m][n] = (f32x4){0.f, 0.f, 0.f, 0.f};
            cur = nxt; cA = nA; cB = nB; ++j; have >>= 16;
            if (wr == 1) BAR;
        }
        WAIT_V(0);
    }
    BAR;
    if (wid == 0) { STAMP(65); STAMPV(67, __builtin_amdgcn_s_memtime()); }
    if (tid == 0) __hip_atomic_fetch_add(ctrl + W_DONE, 1u, __ATOMIC_RELAXED, __HIP_MEMORY_SCOPE_AGENT);
}

void make_order(Params& p) {
    int u = 0, f = 0;
    for (int j = 0; j < 8; ++j) {
        int n = 0, cols[16]; double key[16];
        const int c0 = j == 0 ? 0 : R0 / 4 + (j - 1) * 15, nc = j == 0 ? R0 / 4 : 15, ne = j == 0 ? R0E : GE, no = j == 0 ? R0O : GO;
        for (int col = c0; col < c0 + nc; ++col) {
            if (col == SPLIT_COL || col == SPLIT_COL + 1) continue;
            const int e = (col - c0) * 4;
            double k = e < ne ? (e + 0.5) / ne + ODD_FIRST : (double)(e - ne) / no;
            if (e < ne && e + 3 >= ne) k = 0.0;
            int i = n++;
            while (i > 0 && key[i - 1] > k) { key[i] = key[i - 1]; cols[i] = cols[i - 1]; --i; }
            key[i] = k; cols[i] = col;
        }
        for (int sl = 0; sl < 8; ++sl) {
            if (j == 0) for (int a = 0; a < 16; ++a, ++u) p.blk[u >> 1] |= (unsigned)(a * 8 + sl) << ((u & 1) * 16);
            for (int i = 0; i < n; ++i, ++u) p.blk[u >> 1] |= (unsigned)((16 + cols[i] - (cols[i] > SPLIT_COL ? 2 : 0)) * 8 + sl) << ((u & 1) * 16);
            if (j >= 2) for (int i = 0; i < FWD_MIX; ++i, ++u) p.blk[u >> 1] |= (unsigned)(0x8000 + f++) << ((u & 1) * 16);
        }
    }
    for (; u < NBLK; ++u) p.blk[u >> 1] |= (unsigned)(0x8000 + f++) << ((u & 1) * 16);
}
}

extern "C" void kernel_launch(int rank, int nranks, ncclComm_t comm, hipStream_t stream,
                              void* const* d_in, const int* in_sizes, int n_in,
                              void* d_out, int out_size, const int* in_dtypes, int out_dtype,
                              void* const* d_out_all, void* const* d_ws_all, size_t ws_bytes) {
    if (nranks != NR || (unsigned)rank >= (unsigned)NR || n_in != 2 || in_sizes[0] != M * KL || in_sizes[1] != KL * N
        || out_size != M * N || ws_bytes < WS_NEED) { fprintf(stderr, "unexpected problem shape\n"); return; }
    static bool init = false;
    if (!init) { HIP_CHECK(hipFuncSetAttribute((const void*)fused_gemm, hipFuncAttributeMaxDynamicSharedMemorySize, LDS_BYTES)); init = true; }
    Params p = {};
    p.x = (const float*)d_in[0]; p.w = (const float*)d_in[1]; p.rank = rank;
    for (int r = 0; r < NR; ++r) { p.out[r] = (float*)d_out_all[r]; p.ws[r] = (unsigned char*)d_ws_all[r]; }
    make_order(p);
    hipLaunchKernelGGL(fused_gemm, dim3(GRID), dim3(512), LDS_BYTES, stream, p);
    HIP_CHECK(hipGetLastError());
}
```

```cpp
#include <hip/hip_runtime.h>
#include <rccl/rccl.h>
#include <cstdio>

#define HIP_CHECK(expr) do { hipError_t _e = (expr); if (_e != hipSuccess) { \
    fprintf(stderr, "HIP error %s at %s:%d\n", hipGetErrorString(_e), __FILE__, __LINE__); \
    return; } } while (0)

namespace {
#define LAS __attribute__((address_space(3)))
typedef _Float16 h8 __attribute__((ext_vector_type(8)));
typedef _Float16 h2 __attribute__((ext_vector_type(2)));
typedef float f32x4 __attribute__((ext_vector_type(4)));
typedef unsigned u32x4 __attribute__((ext_vector_type(4)));
typedef __amdgpu_buffer_rsrc_t rsrc_t;

constexpr int NR = 8, M = 4096, N = 32768, KL = 512, NKT = 64, GRID = 256;
constexpr unsigned HTB = 16384, KTB = 32768, PANEL = NKT * KTB;
constexpr unsigned A_BYTES = (M / 256) * PANEL, B_BYTES = (N / 256) * PANEL, BUF_BYTES = A_BYTES + B_BYTES;
constexpr unsigned OUT_BYTES = 512u << 20, X_BYTES = M * KL * 4, W_BYTES = KL * N * 4;
constexpr unsigned SCR_BYTES = GRID * 262144u;
constexpr size_t CTRL_BYTES = 1u << 20, SCR_OFF = CTRL_BYTES + 2 * (size_t)BUF_BYTES, WS_NEED = SCR_OFF + SCR_BYTES;
constexpr int W_DONE = 0, W_TMO = 32, W_PAR = 64, PAR_WORDS = 160 * 32;
constexpr int W_ETA = W_PAR + 2 * PAR_WORDS, W_BOX = W_ETA + 2 * GRID;
constexpr int P_TICKET = 0, P_TILE = 32, P_FWD = 64, P_PRE = 10 * 32, P_SPC = 13 * 32, P_CNT = 16 * 32;
constexpr unsigned CNT_PER_CALL = 64, TILE_PER_CALL = 256, FWD_PER_CALL = 2048;
constexpr unsigned long long DEADLINE = 50000000ull;
constexpr int LDS_BYTES = 8 * HTB + 1024;
constexpr int AUX_SYS = 17, AUX_NT = 2;
#define PUSH_E 2
#define PUSH_O 2
#define POLL_SLEEP 48
#define CONV_GAP 800
#define ODD_FIRST 0.0
#define FWD_MIX 3
#define SPEC_GAP 80
#define NDIR 12
#define K0_BLK 164
#define FWD_FROM 2
#define K1_BLK 200
#define STEAL_OVH 600
#define STEAL_LEAD 3000
#define STEAL_MIN 3
#define END_SLOW 0
#define TAIL_BLK 16
#define UPT 4
#define PUSH_WAVES 2
#define R0E GE
constexpr int GE = 32 - PUSH_E, GO = 32 - PUSH_O, RJ = GE + GO, NPN = 112, R0 = (NPN + 2) * 4 - 7 * RJ, R0O = R0 - R0E, SPLIT_COL = 0;
constexpr int PRE_WGS = 4 * RJ, NPUSH = 4 * (PUSH_E + PUSH_O);
constexpr int NBLK = (16 + NPN) * 8 + FWD_PER_CALL / 8, U_END = NBLK * 8, U_K0 = K0_BLK * 8, U_K1 = K1_BLK * 8, U_K2 = U_END - TAIL_BLK * 8;
constexpr int T_K0 = (U_K0 - NDIR * 8) / UPT, T_K1 = T_K0 + U_K1 - U_K0, T_K2 = T_K1 + (U_K2 - U_K1) / UPT;
static_assert(PUSH_E == PUSH_O, "pushers are workgroups 0..NPUSH-1");
static_assert(FWD_PER_CALL / 8 - 48 * FWD_MIX >= TAIL_BLK, "the tail holds forwarded rows only");
static_assert(GE >= GO && RJ == 60 && R0 % 4 == 0 && R0E <= GE && R0O >= 0 && R0O <= GO, "role split");

struct Params { const float* x; const float* w; float* out[NR]; unsigned char* ws[NR]; int rank; unsigned blk[NBLK / 2]; };

__device__ __forceinline__ rsrc_t mk(const void* p, unsigned bytes) { return __builtin_amdgcn_make_buffer_rsrc((void*)p, 0, bytes, 0x00020000); }
__device__ __forceinline__ unsigned pk(float a, float b) { h2 h; h.x = (_Float16)a; h.y = (_Float16)b; return __builtin_bit_cast(unsigned, h); }
#define WAIT_V(n) asm volatile("s_waitcnt vmcnt(" #n ")" ::: "memory")
#define WAIT_L(n) asm volatile("s_waitcnt lgkmcnt(" #n ")" ::: "memory")
#define WAIT_L0 do { asm volatile("" ::: "memory"); __builtin_amdgcn_s_waitcnt(0xC07F); asm volatile("" ::: "memory"); } while (0)
#define BAR __builtin_amdgcn_s_barrier()
#define SCHED __builtin_amdgcn_sched_barrier(0)
#define STAMP(i) do {} while (0)
#define STAMPV(i, v) do {} while (0)

__device__ __forceinline__ unsigned long long wait_mask(unsigned* word, unsigned thr, unsigned long long need, unsigned* tmo, unsigned call, int slow = 0) {
    const unsigned long long t0 = __builtin_amdgcn_s_memrealtime();
    for (unsigned it = 1;; ++it) {
        const unsigned long long got = __ballot(__hip_atomic_load(word, __ATOMIC_RELAXED, __HIP_MEMORY_SCOPE_SYSTEM) >= thr);
        if ((got & need) == need) return got;
        if ((it & 63) == 0 && (__builtin_amdgcn_readfirstlane(__hip_atomic_load(tmo, __ATOMIC_RELAXED, __HIP_MEMORY_SCOPE_AGENT)) == call
            || __builtin_amdgcn_s_memrealtime() - t0 > DEADLINE)) {
            __hip_atomic_store(tmo, call, __ATOMIC_RELAXED, __HIP_MEMORY_SCOPE_AGENT);
            return 0;
        }
        if (slow) __builtin_amdgcn_s_sleep(POLL_SLEEP); else __builtin_amdgcn_s_sleep(4);
    }
}

__device__ __forceinline__ void conv_load(const Params& p, int it, int s, int sub, int lane, f32x4 (&v)[8]) {
    if (it < 16) {
        const rsrc_t r = mk(p.x, X_BYTES);
        const unsigned a = (unsigned)(it * 256 + sub * 32 + (lane & 15)) * (KL * 4) + s * 256 + (lane >> 4) * 32;
#pragma unroll
        for (int i = 0; i < 8; ++i) v[i] = __builtin_bit_cast(f32x4, __builtin_amdgcn_raw_buffer_load_b128(r, a + (i >> 2) * 16 * (KL * 4) + ((i >> 1) & 1) * 128 + (i & 1) * 16, 0, AUX_NT));
    } else {
        const rsrc_t r = mk(p.w, W_BYTES);
        const unsigned a = (unsigned)(s * 64 + (sub & 1) * 32 + (lane >> 4) * 8) * (N * 4) + (it - 16) * 1024 + (sub >> 1) * 256 + (lane & 15) * 16;
#pragma unroll
        for (int j = 0; j < 8; ++j) v[j] = __builtin_bit_cast(f32x4, __builtin_amdgcn_raw_buffer_load_b128(r, a + j * (N * 4), 0, AUX_NT));
    }
}
__device__ __forceinline__ void conv_store(const Params& p, int it, int s, int sub, int rank, size_t bufoff, int dmask, int rot, int lane, const f32x4 (&v)[8]) {
    u32x4 h[4];
    unsigned o = it * PANEL + (s * 8 + rank) * KTB + lane * 16, s1, s2;
    if (it < 16) {
#pragma unroll
        for (int i = 0; i < 4; ++i) { h[i][0] = pk(v[2 * i][0], v[2 * i][1]); h[i][1] = pk(v[2 * i][2], v[2 * i][3]); h[i][2] = pk(v[2 * i + 1][0], v[2 * i + 1][1]); h[i][3] = pk(v[2 * i + 1][2], v[2 * i + 1][3]); }
        o += sub * 4096; s1 = 1024; s2 = 2048;
    } else {
#pragma unroll
        for (int i = 0; i < 4; ++i) { h[i][0] = pk(v[0][i], v[1][i]); h[i][1] = pk(v[2][i], v[3][i]); h[i][2] = pk(v[4][i], v[5][i]); h[i][3] = pk(v[6][i], v[7][i]); }
        o += (sub >> 1) * 4096 + (sub & 1) * 1024; s1 = 2048; s2 = 16384;
    }
#pragma unroll 1
    for (int q = 0; q < NR; ++q) {
        const int d = (q + rot) & 7;
        if (!((dmask >> d) & 1)) continue;
        const rsrc_t r = mk(p.ws[d] + bufoff, BUF_BYTES);
        __builtin_amdgcn_raw_buffer_store_b128(h[0], r, o, 0, AUX_SYS);
        __builtin_amdgcn_raw_buffer_store_b128(h[1], r, o + s1, 0, AUX_SYS);
        __builtin_amdgcn_raw_buffer_store_b128(h[2], r, o + s2, 0, AUX_SYS);
        __builtin_amdgcn_raw_buffer_store_b128(h[3], r, o + s1 + s2, 0, AUX_SYS);
    }
}
__device__ __forceinline__ int blk_at(const Params& p, int q) { return (p.blk[q >> 1] >> ((q & 1) * 16)) & 0xffff; }

struct Tile { int pm, pn; };
__device__ __forceinline__ bool slot(int j, int g, int x, int rank, Tile& t) {
    if (j >= 8) return false;
    const int e = j == 0 ? ((x & 1) ? R0E : 0) + g : R0 + RJ * (j - 1) + ((x & 1) ? GE : 0) + g, col = e >> 2;
    t.pm = 4 * (x >> 1) + (e & 3);
    t.pn = col < SPLIT_COL ? col : col < SPLIT_COL + 2 ? NPN + (col - SPLIT_COL) * 8 + rank : col - 2;
    return true;
}

#define SA(b, h) (((b) * 2 + (h)) * HTB)
#define SB(b, h) ((4 + (b) * 2 + (h)) * HTB)
#define STAGE(bufoff, soff) do { \
    __builtin_amdgcn_raw_ptr_buffer_load_lds(rAB, (LAS void*)(lds + (bufoff) + ldsw), 16, voff, (soff), 0, AUX_SYS); \
    __builtin_amdgcn_raw_ptr_buffer_load_lds(rAB, (LAS void*)(lds + (bufoff) + ldsw), 16, voff, (soff), 1024, AUX_SYS); } while (0)
#define LDA(dst, b, h) do { _Pragma("unroll") for (int m = 0; m < 4; ++m) _Pragma("unroll") for (int k = 0; k < 2; ++k) dst[m][k] = *(const LAS h8*)(pa + SA(b, h) + m * 2048 + k * 1024); } while (0)
#define LDB(dst, b, h) do { _Pragma("unroll") for (int n = 0; n < 2; ++n) _Pragma("unroll") for (int k = 0; k < 2; ++k) dst[n][k] = *(const LAS h8*)(pb + SA(b, h) + n * 2048 + k * 1024); } while (0)
#define MMA2(ai, Z) do { __builtin_amdgcn_s_setprio(1); SCHED; MMA(ai, 0, At, B0, Z); MMA(ai, 1, At, B1, Z); __builtin_amdgcn_s_setprio(0); } while (0)
#define MMA(ai, bj, At, Bt, Z) do { _Pragma("unroll") for (int m = 0; m < 4; ++m) _Pragma("unroll") for (int n = 0; n < 2; ++n) _Pragma("unroll") for (int k = 0; k < 2; ++k) { \
    acc[ai][bj][m][n] = __builtin_amdgcn_mfma_f32_16x16x32_f16(At[m][k], Bt[n][k], (Z) && k == 0 ? (f32x4){0.f, 0.f, 0.f, 0.f} : acc[ai][bj][m][n], 0, 0, 0); SCHED; } } while (0)

__global__ __launch_bounds__(512, 2) void fused_gemm(Params p) {
    extern __shared__ __attribute__((aligned(16))) unsigned char shm[];
    LAS unsigned char* lds = (LAS unsigned char*)shm;
    const int tid = threadIdx.x, wid = __builtin_amdgcn_readfirstlane(tid >> 6), lane = tid & 63;
    const int x = blockIdx.x & 7, c = blockIdx.x >> 3, rank = p.rank;
    unsigned* ctrl = (unsigned*)p.ws[rank];
    if (wid == 0) { STAMP(64); STAMPV(66, __builtin_amdgcn_s_memtime()); }
    const unsigned long long tstart = __builtin_amdgcn_s_memrealtime();
    const unsigned done = __hip_atomic_load(ctrl + W_DONE, __ATOMIC_RELAXED, __HIP_MEMORY_SCOPE_AGENT);
    const int g = c - ((x & 1) ? PUSH_O : PUSH_E);
    const int q0 = g < 0 ? (x >> 1) * (PUSH_E + PUSH_O) + ((x & 1) ? PUSH_E : 0) + c : g < GO ? g * 8 + x : GO * 8 + (g - GO) * 4 + (x >> 1);
    const int b0 = blk_at(p, q0);
    f32x4 cv[8];
    if (g >= 0) conv_load(p, b0 >> 3, b0 & 7, wid, lane, cv);
    const unsigned call = __builtin_amdgcn_readfirstlane(done) / GRID + 1;
    const unsigned par = call & 1, ncp = (call + 1) >> 1;
    const size_t bufoff = CTRL_BYTES + (size_t)par * BUF_BYTES;
    const int pw = W_PAR + par * PAR_WORDS;
    if (blockIdx.x == 0 && tid == 0) __hip_atomic_store(ctrl + W_PAR + (par ^ 1) * PAR_WORDS + P_TICKET, 0u, __ATOMIC_RELAXED, __HIP_MEMORY_SCOPE_AGENT);
    unsigned char* sigb = p.ws[0];
#pragma unroll
    for (int d = 1; d < NR; ++d) sigb = lane == d ? p.ws[d] : sigb;
    unsigned* const sig = (unsigned*)sigb + pw;

    if (g >= 0) {
        conv_store(p, b0 >> 3, b0 & 7, wid, rank, bufoff, q0 < NDIR ? 0xff : 1 << rank, rank + q0 * 3 + wid * 2 + (wid >> 2), lane, cv);
        WAIT_V(0); BAR;
        if (tid < NR && (q0 < NDIR || tid == rank)) __hip_atomic_fetch_add(sig + P_CNT + (b0 >> 3) * 32 + (b0 & 7), 8u, __ATOMIC_RELAXED, __HIP_MEMORY_SCOPE_SYSTEM);
        if (tid == 64 && q0 >= NDIR) __hip_atomic_fetch_add(ctrl + pw + (q0 < NDIR + 32 ? P_PRE : P_PRE + 32), 1u, __ATOMIC_RELAXED, __HIP_MEMORY_SCOPE_SYSTEM);
        if (wid == 0) STAMP(110);
        const int gi = g - ((x & 1) ? R0O : R0E);
        for (int q = gi < 0 ? 128 : (((x & 1) ? GE - R0E : 0) + gi) * 4 + (x >> 1); q < 128; q += 4 * (RJ - R0)) {
            const int it = 16 + NPN + (q & 15), sl = q >> 4;
            conv_load(p, it, sl, wid, lane, cv); conv_store(p, it, sl, wid, rank, bufoff, 1 << rank, rank, lane, cv);
            WAIT_V(0); BAR;
            if (tid == 0 || (tid == 1 && (q & 7) == rank)) __hip_atomic_fetch_add(ctrl + pw + (tid ? P_CNT + it * 32 + sl : P_SPC), tid ? 8u : 1u, __ATOMIC_RELAXED, __HIP_MEMORY_SCOPE_SYSTEM);
        }
        int word = P_PRE + 64, pass = 1;
        for (int q = q0 + PRE_WGS; q < NBLK; q += PRE_WGS) {
            const int b = blk_at(p, q);
            if (b & 0x8000) continue;
            while (__builtin_amdgcn_s_memrealtime() - tstart < (unsigned)(pass * CONV_GAP + (g & 7) * (CONV_GAP / 8))) __builtin_amdgcn_s_sleep(8);
            conv_load(p, b >> 3, b & 7, wid, lane, cv);
            conv_store(p, b >> 3, b & 7, wid, rank, bufoff, 1 << rank, rank, lane, cv);
            word = lane == pass ? P_CNT + (b >> 3) * 32 + (b & 7) : word;
            ++pass;
        }
        WAIT_V(0); BAR;
        if (tid < pass) __hip_atomic_fetch_add(ctrl + pw + word, lane ? 8u : 1u, __ATOMIC_RELAXED, __HIP_MEMORY_SCOPE_SYSTEM);
        STAMP(100 + wid);
    } else {
        if (wid < PUSH_WAVES) {
            unsigned* const ticket = ctrl + pw + P_TICKET;
            const bool sigl = lane < NR && lane != rank;
            const int rot = wid + q0 * PUSH_WAVES;
            unsigned char* pd[7]; unsigned char* po[7];
#pragma unroll
            for (int q = 0; q < 7; ++q) { const int d = (rank + 1 + (q + rot) % 7) & 7; pd[q] = p.ws[d] + bufoff; po[q] = (unsigned char*)p.out[d]; }
            u32x4 v[4] = {}, h[4];
            unsigned tk = 0;
#define SIGNAL(w) do { if ((w) >= 0 && sigl) __hip_atomic_fetch_add(sig + ((w) >> 4), (unsigned)((w) & 15), __ATOMIC_RELAXED, __HIP_MEMORY_SCOPE_SYSTEM); } while (0)
#define DRAW(n) do { if (lane == 0) asm volatile("global_atomic_add %0, %1, %2, off sc0" : "+v"(tk) : "v"(ticket), "v"((unsigned)(n)) : "memory"); } while (0)
            DRAW(1);
            int stage = 0;
            int u = 0, left = 0, tsz = 1, nfwd = 0, word = -1, pword = -1, ppword = -1;
            unsigned off = 0x80000000u, ps = 1024;
            bool cur_fwd = false, tiles_ok = false;
            WAIT_V(0);
            for (;;) {
                asm volatile("s_waitcnt vmcnt(28)" : "+v"(tk) :: "memory");
                SIGNAL(ppword);
#pragma unroll
                for (int i = 0; i < 4; ++i) h[i] = v[i];
                const bool fresh = left == 0;
                if (fresh) { const int k = __builtin_amdgcn_readfirstlane(tk); const bool big = k < T_K0 || (k >= T_K1 && k < T_K2);
                             u = k < T_K0 ? NDIR * 8 + k * UPT : k < T_K1 ? U_K0 + k - T_K0 : k < T_K2 ? U_K1 + (k - T_K1) * UPT : U_K2 + k - T_K2;
                             left = big ? UPT - 1 : 0; tsz = left + 1; }
                else { ++u; --left; }
                const bool valid = u < U_END;
                const int b = blk_at(p, valid ? u >> 3 : 0), fv = (b & 0xff) * 8 + (u & 7);
                const bool fwd = b & 0x8000;
                const int st = !valid ? 0 : u >= PRE_WGS * 8 ? 3 : u >= (NDIR + 32) * 8 ? 2 : 1;
                if (st > stage) {
                    wait_mask(ctrl + pw + P_PRE + (st - 1) * 32, (st == 3 ? PRE_WGS : st == 2 ? PRE_WGS - 32 - NDIR : 32) * ncp, 1, ctrl + W_TMO, call);
                    stage = st; if (wid == 0) STAMP(110 + st);
                }
                if (fwd && !tiles_ok) { wait_mask(ctrl + pw + P_TILE, TILE_PER_CALL * ncp, 1, ctrl + W_TMO, call); tiles_ok = true; if (wid == 0) STAMP(114); }
                const int nword = left || fwd ? -1 : ((P_CNT + (b >> 3) * 32 + (b & 7)) << 4) + tsz;
                nfwd += fwd;
                const unsigned noff = !fwd ? (b >> 3) * PANEL + ((b & 7) * 8 + rank) * KTB + (u & 7) * 4096
                                           : (unsigned)(fv & 1023) * 4u * (N * 4) + (NPN + (fv >> 10) * 8 + rank) * 1024;
                const unsigned nps = fwd ? N * 4 : 1024;
                if (fresh && valid) DRAW(1);
                const rsrc_t rs = mk(fwd ? (unsigned char*)p.out[rank] : p.ws[rank] + bufoff, fwd ? OUT_BYTES : BUF_BYTES);
#pragma unroll
                for (int i = 0; i < 4; ++i) v[i] = __builtin_amdgcn_raw_buffer_load_b128(rs, noff + lane * 16 + i * nps, 0, AUX_SYS);
                rsrc_t rd[7];
#pragma unroll
                for (int q = 0; q < 7; ++q) rd[q] = mk(cur_fwd ? po[q] : pd[q], cur_fwd ? OUT_BYTES : BUF_BYTES);
#pragma unroll
                for (int i = 0; i < 4; ++i)
#pragma unroll
                    for (int q = 0; q < 7; ++q) __builtin_amdgcn_raw_buffer_store_b128(h[i], rd[q], off + lane * 16 + i * ps, 0, AUX_SYS);
                ppword = pword; pword = word;
                if (!valid) break;
                word = nword; off = noff; ps = nps; cur_fwd = fwd;
            }
            asm volatile("s_waitcnt vmcnt(0)" : "+v"(tk) :: "memory");
            SIGNAL(ppword); SIGNAL(pword);
            if (sigl) __hip_atomic_fetch_add(sig + P_FWD + rank * 32, (unsigned)nfwd, __ATOMIC_RELAXED, __HIP_MEMORY_SCOPE_SYSTEM);
            STAMP(100 + wid);
        }
        if (wid == (PUSH_WAVES > 3 ? 6 : 3)) {
            static_assert(NPUSH == 16, "one pusher per (panel set, unit)");
            const rsrc_t rl = mk(p.ws[rank] + bufoff, BUF_BYTES);
            const int it = 16 + NPN + (q0 & 8);
            wait_mask(ctrl + pw + P_SPC, 128 * ncp, 1, ctrl + W_TMO, call);
            for (int sl = 0; sl < 8; ++sl) {
                const unsigned off = it * PANEL + (sl * 8 + rank) * KTB + (q0 & 7) * 4096 + lane * 16;
                u32x4 z[7][4];
#pragma unroll
                for (int q = 0; q < 7; ++q)
#pragma unroll
                    for (int i = 0; i < 4; ++i) z[q][i] = __builtin_amdgcn_raw_buffer_load_b128(rl, off + ((rank + 1 + (q + q0) % 7) & 7) * PANEL + i * 1024, 0, AUX_SYS);
                wait_mask(ctrl + pw + P_TICKET, (sl * 16 + q0) * T_K0 / 128, 1, ctrl + W_TMO, call, 1);
#pragma unroll
                for (int i = 0; i < 4; ++i)
#pragma unroll
                    for (int q = 0; q < 7; ++q) { const int d = (rank + 1 + (q + q0) % 7) & 7; __builtin_amdgcn_raw_buffer_store_b128(z[q][i], mk(p.ws[d] + bufoff, BUF_BYTES), off + d * PANEL + i * 1024, 0, AUX_SYS); }
                WAIT_V(0);
                if (lane < NR && lane != rank) __hip_atomic_fetch_add(sig + P_CNT + (it + lane) * 32 + sl, 1u, __ATOMIC_RELAXED, __HIP_MEMORY_SCOPE_SYSTEM);
            }
            STAMP(102);
        }
    }
    if (blockIdx.x == 0 && wid == 4) {
        unsigned long long* const tab = (unsigned long long*)(ctrl + W_ETA);
        LAS unsigned* const kb = (LAS unsigned*)lds; LAS unsigned* const ord = kb + GRID;
        unsigned key[4]; bool all = false;
        while (!all && __builtin_amdgcn_s_memrealtime() - tstart < (1u << 20)) {
            __builtin_amdgcn_s_sleep(127); __builtin_amdgcn_s_sleep(127);
            int n = 0;
#pragma unroll
            for (int i = 0; i < 4; ++i) {
                const unsigned d = (unsigned)__hip_atomic_load(tab + i * 64 + lane, __ATOMIC_RELAXED, __HIP_MEMORY_SCOPE_AGENT) - (unsigned)tstart;
                key[i] = d < (1u << 21) && i * 64 + lane >= NPUSH ? (d << 8) + i * 64 + lane : 0u;
                n += __popcll(__ballot(key[i] != 0));
            }
            all = n == PRE_WGS;
        }
        if (all) {
            int r[4] = {0, 0, 0, 0};
#pragma unroll
            for (int i = 0; i < 4; ++i) kb[i * 64 + lane] = key[i];
            for (int k = NPUSH; k < GRID; ++k) { const unsigned o = kb[k];
#pragma unroll
                for (int i = 0; i < 4; ++i) r[i] += o < key[i]; }
#pragma unroll
            for (int i = 0; i < 4; ++i) if (key[i]) ord[r[i]] = key[i];
            const unsigned cap = (unsigned)(__builtin_amdgcn_s_memrealtime() - tstart) + 30000u, first = (ord[0] >> 8) < cap ? ord[0] >> 8 : cap;
            while (__builtin_amdgcn_s_memrealtime() - tstart + STEAL_LEAD < first) __builtin_amdgcn_s_sleep(32);
            for (int k = lane; k < PRE_WGS / 2; k += 64) {
                const int th = ord[k] & 0xff, v = ord[PRE_WGS - 1 - k] & 0xff;
                const unsigned long long et = __hip_atomic_load(tab + th, __ATOMIC_RELAXED, __HIP_MEMORY_SCOPE_AGENT), ev = __hip_atomic_load(tab + v, __ATOMIC_RELAXED, __HIP_MEMORY_SCOPE_AGENT);
                const int rem = (int)((unsigned)ev + (unsigned)(ev >> 48) - (unsigned)et - (unsigned)(et >> 48)) - STEAL_OVH;
                const int n = rem > 0 ? rem / (int)(((ev >> 32) & 0xffff) + ((et >> 32) & 0xffff)) : 0;
                if (n >= STEAL_MIN && n <= NKT / 4)
                    __hip_atomic_store((unsigned long long*)(ctrl + W_BOX + th * 32 + 2), ((unsigned long long)call << 32) | (call << 16) | (v << 8) | (NKT - 2 * n), __ATOMIC_RELAXED, __HIP_MEMORY_SCOPE_AGENT);
            }
        }
    }
    if (blockIdx.x == 1 && wid == 5) {
        if (!wait_mask(ctrl + pw + P_FWD + (lane & 7) * 32, FWD_PER_CALL * ncp, 0xff & ~(1 << rank), ctrl + W_TMO, call, END_SLOW) && lane == 0) p.out[rank][0] = __builtin_nanf("");
        __builtin_amdgcn_fence(__ATOMIC_ACQUIRE, "");
    }
    if (g >= 0) {
        const int wr = wid >> 2, wc = wid & 3, fr = lane & 15, fq = lane >> 4;
        const rsrc_t rAB = mk(p.ws[rank] + bufoff, BUF_BYTES), rCt = mk(ctrl, (unsigned)CTRL_BYTES);
        const unsigned voff = wid * 2048 + lane * 16, ldsw = wid * 2048;
        LAS unsigned char* pa = lds + wr * 8192 + lane * 16; LAS unsigned char* pb = lds + SB(0, 0) + wc * 4096 + lane * 16;
        asm volatile("" : "+v"(pa), "+v"(pb));
        const unsigned thr = CNT_PER_CALL * ncp;
        volatile LAS unsigned* const sm = (volatile LAS unsigned*)(lds + 8 * HTB);
        int j = g < ((x & 1) ? R0O : R0E) ? 0 : 1, t0 = 0; Tile cur, nxt;
        slot(j, g, x, rank, cur);
        bool bad = false, helping = false;
        unsigned tjob = (unsigned)tstart, pace = 0, eta6 = 0;
        unsigned have = 0;
        if (tid == 0) { sm[128] = NKT; sm[65] = 0; sm[67] = 0; }
        if (wid == 0) {
            const int ln = (lane & 8) ? cur.pm : 16 + cur.pn;
            have = (unsigned)wait_mask(ctrl + pw + P_CNT + ln * 32 + (lane & 7), thr, 0x101, ctrl + W_TMO, call, 1) & 0xffffu;
            bad = have == 0;
            STAMP(68);
        }
        asm volatile("s_waitcnt lgkmcnt(0)" ::: "memory"); BAR; asm volatile("" ::: "memory");
        unsigned long long wsum = 0; (void)wsum;
        f32x4 acc[2][2][4][2];
        h8 At[4][2], B0[2][2], B1[2][2];
        unsigned cA = cur.pm * PANEL, cB = A_BYTES + cur.pn * PANEL;
        STAGE(SB(0, 0), cB); STAGE(SB(0, 1), cB + HTB); STAGE(SA(0, 0), cA); STAGE(SA(0, 1), cA + HTB);
        if (wr == 1) BAR;
        WAIT_V(2); BAR;
        STAGE(SB(1, 0), cB + KTB); STAGE(SA(1, 0), cA + KTB); STAGE(SB(1, 1), cB + HTB + KTB);
        WAIT_V(6); BAR;
        for (;;) {
            bool has_next = slot(j + 1, g, x, rank, nxt);
            const bool final = j == 7;
            if (!has_next) nxt = cur;
            unsigned nA = nxt.pm * PANEL, nB = A_BYTES + nxt.pn * PANEL;
            const int sel = (lane >> 3) & 3, ln = sel == 0 ? 16 + cur.pn : sel == 1 ? cur.pm : sel == 2 ? 16 + nxt.pn : nxt.pm;
            unsigned fw = pw + P_CNT + ln * 32 + (lane & 7), need_next = 0x01010000u;
            if (wid == 0) {
                const unsigned now = (unsigned)__builtin_amdgcn_s_memrealtime(), dur = now - tjob;
                tjob = now; pace = dur / (NKT / 2);
                if (j == 6) eta6 = now + 2 * dur;
                const unsigned corr = (final ? now + dur - eta6 : 0u) + 0x8000u;
                if ((j == 6 || final) && lane == 0) __hip_atomic_store((unsigned long long*)(ctrl + W_ETA) + blockIdx.x,
                    ((unsigned long long)((corr << 16) | (pace & 0xffffu)) << 32) | eta6, __ATOMIC_RELAXED, __HIP_MEMORY_SCOPE_AGENT);
                __builtin_amdgcn_s_waitcnt(0xC07F);
            }
            int plan = NKT;
            auto arrival = [&](int t) __attribute__((always_inline)) {
                if (wid == 0 && (t & 4)) {
                    const unsigned need = t >= NKT - 4 ? need_next : 0x202u << (t >> 3);
                    if ((have & need) != need && !helping) {
                        if (!(t & 2)) __builtin_amdgcn_raw_ptr_buffer_load_lds(rCt, (LAS void*)(lds + 8 * HTB), 4, fw * 4, 0, 0, AUX_SYS);
                        else {
                            have = (unsigned)__ballot(sm[lane] >= thr);
                            if ((have & need) != need) {
                                have = (unsigned)wait_mask(ctrl + fw, thr, need, ctrl + W_TMO, call, 1);
                                bad |= have == 0;
                            }
                            asm volatile("" ::: "memory");
                        }
                    }
                }
            };
#define TRIP(Z, HOOK) do { \
                const unsigned a1 = cA + (t + 1) * KTB; \
                const unsigned a2 = t == NKT - 2 ? nA : cA + (t + 2) * KTB, b2 = t == NKT - 2 ? nB : cB + (t + 2) * KTB; \
                const unsigned a3 = a2 + KTB, b3 = b2 + KTB; \
                LDB(B0, 0, 0); LDB(B1, 0, 1); SCHED; LDA(At, 0, 0); STAGE(SA(1, 1), a1 + HTB); \
                WAIT_V(8); WAIT_L0; BAR; MMA2(0, Z); BAR; SCHED; \
                LDA(At, 0, 1); STAGE(SB(0, 0), b2); STAGE(SB(0, 1), b2 + HTB); STAGE(SA(0, 0), a2); \
                WAIT_V(8); WAIT_L0; BAR; MMA2(1, Z); BAR; SCHED; \
                LDB(B0, 1, 0); LDB(B1, 1, 1); SCHED; LDA(At, 1, 0); STAGE(SA(0, 1), a2 + HTB); \
                HOOK \
                WAIT_V(8); WAIT_L0; BAR; MMA2(0, false); BAR; SCHED; \
                LDA(At, 1, 1); STAGE(SB(1, 0), b3); STAGE(SB(1, 1), b3 + HTB); STAGE(SA(1, 0), a3); \
                WAIT_V(8); WAIT_L0; BAR; MMA2(1, false); BAR; SCHED; } while (0)
            volatile LAS unsigned* pvp = sm + (lane < 4 ? 64 + lane : 128);
            int t = t0;
            TRIP(true, ); t += 2;
            for (const int te = final ? NKT / 2 - 12 : NKT; t < te; t += 2) { arrival(t); TRIP(false, ); }
            for (; t < (plan & 0xff); t += 2) {
                if (t == NKT - 2 && (plan >> 16)) {
                    const int v = (plan >> 8) & 0xff;
                    slot(7, (v >> 3) - ((v & 1) ? PUSH_O : PUSH_E), v & 7, rank, nxt);
                    has_next = true; nA = nxt.pm * PANEL + (plan >> 16) * KTB; nB = A_BYTES + nxt.pn * PANEL + (plan >> 16) * KTB;
                }
                if (t == NKT - 4 && need_next != 0x01010000u) have &= 0xffffu;
                arrival(t);
                unsigned pv = 0;
                const bool look = !(t & 2) && plan == NKT;
                if (look) pv = *pvp;
                auto hook = [&]() __attribute__((always_inline)) {
                    if (!look) return;
                    plan = __builtin_amdgcn_readlane(pv, 4);
                    if (wid == 0 && plan == NKT) {
                        const unsigned of = __builtin_amdgcn_readlane(pv, 0), as = __builtin_amdgcn_readlane(pv, 2), tag = call & 0xffffu;
                        if (__builtin_amdgcn_readlane(pv, 1) == call && (of >> 16) == tag && (int)(of & 0xff) >= t + 6) { if (lane == 0) sm[128] = of & 0xffff; }
                        else if (__builtin_amdgcn_readlane(pv, 3) == call && (as >> 16) == tag && t <= NKT - 8) {
                            const int v = (as >> 8) & 0xff;
                            if (lane == 0) {
                                sm[128] = ((as & 0xff) << 16) | (v << 8) | NKT;
                                __hip_atomic_store((unsigned long long*)(ctrl + W_BOX + v * 32), ((unsigned long long)call << 32) | (tag << 16) | (blockIdx.x << 8) | (as & 0xff), __ATOMIC_RELAXED, __HIP_MEMORY_SCOPE_AGENT);
                            }
                            Tile vt; slot(7, (v >> 3) - ((v & 1) ? PUSH_O : PUSH_E), v & 7, rank, vt);
                            if (lane & 16) fw = pw + P_CNT + ((lane & 8) ? vt.pm : 16 + vt.pn) * 32 + (lane & 7);
                            have &= 0xffffu; need_next = 0xffff0000u;
                            STAMP(71); STAMPV(70, as & 0xffff);
                        }
                        __builtin_amdgcn_raw_ptr_buffer_load_lds(rCt, (LAS void*)(lds + 8 * HTB + 256), 4, (lane & 3) * 4, (W_BOX + blockIdx.x * 32) * 4, 0, AUX_SYS);
                    }
                };
                TRIP(false, hook(););
            }
            if (wid == 0 && !helping) { STAMP(72 + j); STAMPV(88 + j, wsum); wsum = 0; STAMPV(116 + j, __builtin_amdgcn_s_memtime()); }
            if (wr == 0) BAR;
            const rsrc_t rS = mk(p.ws[rank] + SCR_OFF, SCR_BYTES);
            const unsigned sb = (unsigned)(wr * 64 + 4 * fq) * 1024 + (wc * 64 + 4 * fr) * 4;
            if ((plan & 0xff) < NKT) {
                const unsigned hb = ((plan >> 8) & 0xff) * 262144u;
                if (wid == 0) { bad |= !wait_mask(ctrl + W_BOX + ((plan >> 8) & 0xff) * 32 + 4, call, 1, ctrl + W_TMO, call); STAMP(69); }
                WAIT_V(0); BAR;
                f32x4 z[16];
#pragma unroll
                for (int i = 0; i < 16; ++i)
                    __builtin_amdgcn_raw_ptr_buffer_load_lds(rS, (LAS void*)(lds + wid * 16384 + i * 1024), 16, sb, hb + ((i >> 2) * 16 + (i & 3)) * 1024, 0, AUX_SYS);
#pragma unroll
                for (int i = 0; i < 16; ++i) z[i] = __builtin_bit_cast(f32x4, __builtin_amdgcn_raw_buffer_load_b128(rS, sb, hb + (128 + (i >> 2) * 16 + (i & 3)) * 1024, AUX_SYS));
                WAIT_V(0);
#pragma unroll
                for (int ai = 1; ai >= 0; --ai)
#pragma unroll
                    for (int i = 0; i < 16; ++i) {
                        if (ai == 0) z[i] = *(const LAS f32x4*)(lds + wid * 16384 + i * 1024 + lane * 16);
                        acc[ai][0][i >> 2][0][i & 3] += z[i][0]; acc[ai][0][i >> 2][1][i & 3] += z[i][1]; acc[ai][1][i >> 2][0][i & 3] += z[i][2]; acc[ai][1][i >> 2][1][i & 3] += z[i][3];
                    }
            }
            if (bad) acc[0][0][0][0][0] = __builtin_nanf("");
#define C_AUX AUX_NT
#define FINAL_WT 0
            const bool own = cur.pn >= NPN || helping || (FINAL_WT && final);
            const rsrc_t rC = helping ? rS : mk(p.out[rank], OUT_BYTES);
            const unsigned pitch = helping ? 1024 : N * 4;
            const unsigned cb = helping ? blockIdx.x * 262144u + sb : (unsigned)(cur.pm * 256 + wr * 64 + 4 * fq) * (N * 4) + cur.pn * 1024 + (wc * 64 + 4 * fr) * 4;
#pragma unroll
            for (int ai = 0; ai < 2; ++ai)
#pragma unroll
                for (int m = 0; m < 4; ++m)
#pragma unroll
                    for (int q = 0; q < 4; ++q) {
                        const f32x4 o = {acc[ai][0][m][0][q], acc[ai][0][m][1][q], acc[ai][1][m][0][q], acc[ai][1][m][1][q]};
                        const unsigned co = cb + (unsigned)(ai * 128 + m * 16 + q) * pitch;
                        if (own) __builtin_amdgcn_raw_buffer_store_b128(__builtin_bit_cast(u32x4, o), rC, co, 0, AUX_SYS);
                        else __builtin_amdgcn_raw_buffer_store_b128(__builtin_bit_cast(u32x4, o), rC, co, 0, C_AUX);
                    }
            if (cur.pn >= NPN && !helping) {
                WAIT_V(0);
                if (lane == 0) __hip_atomic_fetch_add(ctrl + pw + P_TILE, 1u, __ATOMIC_RELAXED, __HIP_MEMORY_SCOPE_AGENT);
            }
            if (wid == 0 && !helping) STAMP(80 + j);
            if (!has_next) break;
            t0 = plan >> 16; helping = final;
            cur = nxt; cA = nA - t0 * KTB; cB = nB - t0 * KTB; ++j; have >>= 16;
            if (wr == 1) BAR;
        }
        WAIT_V(0);
        if (helping) {
            BAR;
            if (tid == 0) __hip_atomic_store(ctrl + W_BOX + blockIdx.x * 32 + 4, call, __ATOMIC_RELAXED, __HIP_MEMORY_SCOPE_AGENT);
        }
    }
    BAR;
    if (wid == 0) { STAMP(65); STAMPV(67, __builtin_amdgcn_s_memtime()); }
    if (tid == 0) __hip_atomic_fetch_add(ctrl + W_DONE, 1u, __ATOMIC_RELAXED, __HIP_MEMORY_SCOPE_AGENT);
}

void make_order(Params& p) {
    int u = 0, f = 0;
    for (int j = 0; j < 8; ++j) {
        int n = 0, cols[16]; double key[16];
        const int c0 = j == 0 ? 0 : R0 / 4 + (j - 1) * 15, nc = j == 0 ? R0 / 4 : 15, ne = j == 0 ? R0E : GE, no = j == 0 ? R0O : GO;
        for (int col = c0; col < c0 + nc; ++col) {
            if (col == SPLIT_COL || col == SPLIT_COL + 1) continue;
            const int e = (col - c0) * 4;
            double k = e < ne ? (e + 0.5) / ne + ODD_FIRST : (double)(e - ne) / no;
            if (e < ne && e + 3 >= ne) k = 0.0;
            int i = n++;
            while (i > 0 && key[i - 1] > k) { key[i] = key[i - 1]; cols[i] = cols[i - 1]; --i; }
            key[i] = k; cols[i] = col;
        }
        for (int sl = 0; sl < 8; ++sl) {
            if (j == 0) for (int a = 0; a < 16; ++a, ++u) p.blk[u >> 1] |= (unsigned)(a * 8 + sl) << ((u & 1) * 16);
            for (int i = 0; i < n; ++i, ++u) p.blk[u >> 1] |= (unsigned)((16 + cols[i] - (cols[i] > SPLIT_COL ? 2 : 0)) * 8 + sl) << ((u & 1) * 16);
            if (j >= FWD_FROM) for (int i = 0; i < FWD_MIX; ++i, ++u) p.blk[u >> 1] |= (unsigned)(0x8000 + f++) << ((u & 1) * 16);
        }
    }
    for (; u < NBLK; ++u) p.blk[u >> 1] |= (unsigned)(0x8000 + f++) << ((u & 1) * 16);
}
}

extern "C" void kernel_launch(int rank, int nranks, ncclComm_t comm, hipStream_t stream,
                              void* const* d_in, const int* in_sizes, int n_in,
                              void* d_out, int out_size, const int* in_dtypes, int out_dtype,
                              void* const* d_out_all, void* const* d_ws_all, size_t ws_bytes) {
    if (nranks != NR || (unsigned)rank >= (unsigned)NR || n_in != 2 || in_sizes[0] != M * KL || in_sizes[1] != KL * N
        || out_size != M * N || ws_bytes < WS_NEED) { fprintf(stderr, "unexpected problem shape\n"); return; }
    static bool init = false;
    if (!init) { HIP_CHECK(hipFuncSetAttribute((const void*)fused_gemm, hipFuncAttributeMaxDynamicSharedMemorySize, LDS_BYTES)); init = true; }
    Params p = {};
    p.x = (const float*)d_in[0]; p.w = (const float*)d_in[1]; p.rank = rank;
    for (int r = 0; r < NR; ++r) { p.out[r] = (float*)d_out_all[r]; p.ws[r] = (unsigned char*)d_ws_all[r]; }
    make_order(p);
    hipLaunchKernelGGL(fused_gemm, dim3(GRID), dim3(512), LDS_BYTES, stream, p);
    HIP_CHECK(hipGetLastError());
}
```

```cpp
#include <hip/hip_runtime.h>
#include <rccl/rccl.h>
#include <cstdio>

#define HIP_CHECK(expr) do { hipError_t _e = (expr); if (_e != hipSuccess) { \
    fprintf(stderr, "HIP error %s at %s:%d\n", hipGetErrorString(_e), __FILE__, __LINE__); \
    return; } } while (0)

namespace {
#define LAS __attribute__((address_space(3)))
typedef _Float16 h8 __attribute__((ext_vector_type(8)));
typedef _Float16 h2 __attribute__((ext_vector_type(2)));
typedef float f32x4 __attribute__((ext_vector_type(4)));
typedef unsigned u32x4 __attribute__((ext_vector_type(4)));
typedef __amdgpu_buffer_rsrc_t rsrc_t;

constexpr int NR = 8, M = 4096, N = 32768, KL = 512, NKT = 64, GRID = 256;
constexpr unsigned HTB = 16384, KTB = 32768, PANEL = NKT * KTB;
constexpr unsigned A_BYTES = (M / 256) * PANEL, B_BYTES = (N / 256) * PANEL, BUF_BYTES = A_BYTES + B_BYTES;
constexpr unsigned OUT_BYTES = 512u << 20, X_BYTES = M * KL * 4, W_BYTES = KL * N * 4;
constexpr unsigned SCR_BYTES = GRID * 262144u;
constexpr size_t CTRL_BYTES = 1u << 20, SCR_OFF = CTRL_BYTES + 2 * (size_t)BUF_BYTES, WS_NEED = SCR_OFF + SCR_BYTES;
constexpr int W_DONE = 0, W_TMO = 32, W_PAR = 64, PAR_WORDS = 160 * 32;
constexpr int W_ETA = W_PAR + 2 * PAR_WORDS, W_BOX = W_ETA + 2 * GRID;
constexpr int W_AMAX = W_BOX + GRID * 32, W_AIN = W_AMAX + 32, W_DEC = W_AIN + 128;
constexpr int P_TICKET = 0, P_TILE = 32, P_FWD = 64, P_PRE = 10 * 32, P_SPC = 13 * 32, P_CNT = 16 * 32;
constexpr unsigned CNT_PER_CALL = 128, TILE_PER_CALL = 256, FWD_PER_CALL = 2048;
constexpr unsigned long long DEADLINE = 50000000ull;
constexpr int LDS_BYTES = 8 * HTB + 1024;
constexpr int AUX_SYS = 17, AUX_NT = 2;
#define PUSH_E 2
#define PUSH_O 2
#define POLL_SLEEP 48
#define CONV_GAP 500
#define FWD_MIX 30
#define NDIR 20
#define K0_BLK ((14 + (R0 + 3) / 4) * 8 - 36)
#define FWD_FROM 2
#define K1_BLK ((14 + (R0 + 3) / 4) * 8 + 40)
#define STEAL_OVH 600
#define STEAL_LEAD 3000
#define STEAL_MIN 3
#define TAIL_BLK 16
#define XLEAD 0.0
#define FWD_RUN 1
#define SKEW 18.0
#define LAZY 2
#define WUPT 2
#define UPT 8
#define PUSH_WAVES 1
#define R0E 24
constexpr int GE = 32 - PUSH_E, GO = 32 - PUSH_O, RJ = GE + GO, NPN = 112, R0 = (NPN + 2) * 4 - 7 * RJ, R0O = R0 - R0E, SPLIT_COL = 0;
constexpr int PRE_WGS = 4 * RJ, NPUSH = 4 * (PUSH_E + PUSH_O);
constexpr int NBLK = (16 + NPN) * 8 + FWD_PER_CALL / 8, U_END = NBLK * 8, U_K0 = K0_BLK * 8, U_K1 = K1_BLK * 8, U_K2 = U_END - TAIL_BLK * 8;
constexpr int T_K0 = (U_K0 - NDIR * 8) / UPT, T_K1 = T_K0 + (U_K1 - U_K0) / WUPT, T_K2 = T_K1 + (U_K2 - U_K1) / UPT;
static_assert(PUSH_E >= 1 && PUSH_O >= 1, "workgroups 0 and 1 are pushers");
static_assert(FWD_PER_CALL / 8 - 48 * FWD_MIX / 10 >= TAIL_BLK, "the tail holds forwarded rows only");
static_assert(GE >= GO && R0E <= GE && R0O >= 0 && R0O <= GO, "role split");

struct Params { const float* x; const float* w; float* out[NR]; unsigned char* ws[NR]; int rank; unsigned blk[NBLK / 2]; };

__device__ __forceinline__ rsrc_t mk(const void* p, unsigned bytes) { return __builtin_amdgcn_make_buffer_rsrc((void*)p, 0, bytes, 0x00020000); }
__device__ __forceinline__ unsigned pk(float a, float b) { h2 h; h.x = (_Float16)a; h.y = (_Float16)b; return __builtin_bit_cast(unsigned, h); }
#define WAIT_V(n) asm volatile("s_waitcnt vmcnt(" #n ")" ::: "memory")
#define WAIT_L(n) asm volatile("s_waitcnt lgkmcnt(" #n ")" ::: "memory")
#define WAIT_L0 do { asm volatile("" ::: "memory"); __builtin_amdgcn_s_waitcnt(0xC07F); asm volatile("" ::: "memory"); } while (0)
#define BAR __builtin_amdgcn_s_barrier()
#define SCHED __builtin_amdgcn_sched_barrier(0)

__device__ __forceinline__ unsigned long long wait_mask(unsigned* word, unsigned thr, unsigned long long need, unsigned* tmo, unsigned call, int slow = 0) {
    const unsigned long long t0 = __builtin_amdgcn_s_memrealtime();
    for (unsigned it = 1;; ++it) {
        const unsigned long long got = __ballot(__hip_atomic_load(word, __ATOMIC_RELAXED, __HIP_MEMORY_SCOPE_SYSTEM) >= thr);
        if ((got & need) == need) return got;
        if ((it & 63) == 0 && (__builtin_amdgcn_readfirstlane(__hip_atomic_load(tmo, __ATOMIC_RELAXED, __HIP_MEMORY_SCOPE_AGENT)) == call
            || __builtin_amdgcn_s_memrealtime() - t0 > DEADLINE)) {
            __hip_atomic_store(tmo, call, __ATOMIC_RELAXED, __HIP_MEMORY_SCOPE_AGENT);
            return 0;
        }
        if (slow) __builtin_amdgcn_s_sleep(POLL_SLEEP); else __builtin_amdgcn_s_sleep(4);
        for (int i = 1; i < slow; ++i) __builtin_amdgcn_s_sleep(127);
    }
}

__device__ __forceinline__ void conv_load(const Params& p, int it, int s, int sub, int lane, f32x4 (&v)[8]) {
    if (it < 16) {
        const rsrc_t r = mk(p.x, X_BYTES);
        const unsigned a = (unsigned)(it * 256 + sub * 32 + (lane & 15)) * (KL * 4) + s * 256 + (lane >> 4) * 32;
#pragma unroll
        for (int i = 0; i < 8; ++i) v[i] = __builtin_bit_cast(f32x4, __builtin_amdgcn_raw_buffer_load_b128(r, a + (i >> 2) * 16 * (KL * 4) + ((i >> 1) & 1) * 128 + (i & 1) * 16, 0, AUX_NT));
    } else {
        const rsrc_t r = mk(p.w, W_BYTES);
        const unsigned a = (unsigned)(s * 64 + (sub & 1) * 32 + (lane >> 4) * 8) * (N * 4) + (it - 16) * 1024 + (sub >> 1) * 256 + (lane & 15) * 16;
#pragma unroll
        for (int j = 0; j < 8; ++j) v[j] = __builtin_bit_cast(f32x4, __builtin_amdgcn_raw_buffer_load_b128(r, a + j * (N * 4), 0, AUX_NT));
    }
}
__device__ __forceinline__ void conv_store(const Params& p, int it, int s, int sub, int rank, size_t bufoff, int dmask, int rot, int lane, f32x4 (&v)[8], int exw, float (&amax)[2]) {
    u32x4 h[4];
    float am = 0.f;
    const int e = it < 16 ? exw >> 16 : (short)exw;
#pragma unroll
    for (int i = 0; i < 8; ++i)
#pragma unroll
        for (int k = 0; k < 4; ++k) { am = fmaxf(am, fabsf(v[i][k])); v[i][k] = __builtin_amdgcn_ldexpf(v[i][k], e); }
    if (it < 16) amax[0] = fmaxf(amax[0], am); else amax[1] = fmaxf(amax[1], am);
    unsigned o = it * PANEL + (s * 8 + rank) * KTB + lane * 16, s1, s2;
    if (it < 16) {
#pragma unroll
        for (int i = 0; i < 4; ++i) { h[i][0] = pk(v[2 * i][0], v[2 * i][1]); h[i][1] = pk(v[2 * i][2], v[2 * i][3]); h[i][2] = pk(v[2 * i + 1][0], v[2 * i + 1][1]); h[i][3] = pk(v[2 * i + 1][2], v[2 * i + 1][3]); }
        o += sub * 4096; s1 = 1024; s2 = 2048;
    } else {
#pragma unroll
        for (int i = 0; i < 4; ++i) { h[i][0] = pk(v[0][i], v[1][i]); h[i][1] = pk(v[2][i], v[3][i]); h[i][2] = pk(v[4][i], v[5][i]); h[i][3] = pk(v[6][i], v[7][i]); }
        o += (sub >> 1) * 4096 + (sub & 1) * 1024; s1 = 2048; s2 = 16384;
    }
#pragma unroll 1
    for (int q = 0; q < NR; ++q) {
        const int d = (q + rot) & 7;
        if (!((dmask >> d) & 1)) continue;
        const rsrc_t r = mk(p.ws[d] + bufoff, BUF_BYTES);
        __builtin_amdgcn_raw_buffer_store_b128(h[0], r, o, 0, AUX_SYS);
        __builtin_amdgcn_raw_buffer_store_b128(h[1], r, o + s1, 0, AUX_SYS);
        __builtin_amdgcn_raw_buffer_store_b128(h[2], r, o + s2, 0, AUX_SYS);
        __builtin_amdgcn_raw_buffer_store_b128(h[3], r, o + s1 + s2, 0, AUX_SYS);
    }
}
__device__ __forceinline__ int blk_at(const Params& p, int q) { return (p.blk[q >> 1] >> ((q & 1) * 16)) & 0xffff; }

struct Tile { int pm, pn; };
__device__ __forceinline__ bool slot(int j, int g, int x, int rank, Tile& t) {
    if (j >= 8) return false;
    const int e = j == 0 ? ((x & 1) ? R0 - GO : R0E - GE) + g : R0 + RJ * (j - 1) + ((x & 1) ? GE : 0) + g, col = e >> 2;
    t.pm = 4 * (x >> 1) + (e & 3);
    t.pn = col < SPLIT_COL ? col : col < SPLIT_COL + 2 ? NPN + (col - SPLIT_COL) * 8 + rank : col - 2;
    return true;
}

#define SA(b, h) (((b) * 2 + (h)) * HTB)
#define SB(b, h) ((4 + (b) * 2 + (h)) * HTB)
#define STAGE(bufoff, soff) do { \
    __builtin_amdgcn_raw_ptr_buffer_load_lds(rAB, (LAS void*)(lds + (bufoff) + ldsw), 16, voff, (soff), 0, AUX_SYS); \
    __builtin_amdgcn_raw_ptr_buffer_load_lds(rAB, (LAS void*)(lds + (bufoff) + ldsw), 16, voff, (soff), 1024, AUX_SYS); } while (0)
#define LDA(dst, b, h) do { _Pragma("unroll") for (int m = 0; m < 4; ++m) _Pragma("unroll") for (int k = 0; k < 2; ++k) dst[m][k] = *(const LAS h8*)(pa + SA(b, h) + m * 2048 + k * 1024); } while (0)
#define LDB(dst, b, h) do { _Pragma("unroll") for (int n = 0; n < 2; ++n) _Pragma("unroll") for (int k = 0; k < 2; ++k) dst[n][k] = *(const LAS h8*)(pb + SA(b, h) + n * 2048 + k * 1024); } while (0)
#define AIDX(ai, bj, m, n) (((((ai) * 2 + (bj)) * 4 + (m)) * 2 + (n)) * 4)
template <int I, bool Z> __device__ __forceinline__ void mf(const h8& a, const h8& b) {
    if (Z) asm volatile("v_mfma_f32_16x16x32_f16 a[%c2:%c3], %0, %1, 0" :: "v"(a), "v"(b), "i"(I), "i"(I + 3));
    else asm volatile("v_mfma_f32_16x16x32_f16 a[%c2:%c3], %0, %1, a[%c2:%c3]" :: "v"(a), "v"(b), "i"(I), "i"(I + 3));
}
#define ORD 5
template <int AI, bool Z, int I> __device__ __forceinline__ void mma(const h8 (&At)[4][2], const h8 (&B0)[2][2], const h8 (&B1)[2][2]) {
    if constexpr (I < 16) {
        constexpr int o = I >> 2, i = (o & 1) ? 3 - (I & 3) : I & 3, m = ORD >= 8 ? i : o, n4 = ORD >= 8 ? o : i, k = (ORD == 5 || ORD == 9) ? I & 1 : 0;
        mf<AIDX(AI, n4 >> 1, m, n4 & 1), Z>(At[m][k], n4 < 2 ? B0[n4 & 1][k] : B1[n4 & 1][k]);
        mf<AIDX(AI, n4 >> 1, m, n4 & 1), false>(At[m][k ^ 1], n4 < 2 ? B0[n4 & 1][k ^ 1] : B1[n4 & 1][k ^ 1]);
        mma<AI, Z, I + 1>(At, B0, B1);
    }
}
#define MMA2(ai, Z) do { __builtin_amdgcn_s_setprio(1); SCHED; mma<ai, Z, 0>(At, B0, B1); SCHED; __builtin_amdgcn_s_setprio(0); } while (0)
template <int I> __device__ __forceinline__ float ard() { float r; asm volatile("v_accvgpr_read_b32 %0, a%c1" : "=v"(r) : "i"(I)); return r; }
template <int I, int I1, int AUXC, bool ADD> __device__ __forceinline__ void stores(rsrc_t rC, unsigned cb, unsigned pitch, const f32x4 (&z)[16]) {
    if constexpr (I < I1) {
        constexpr int ai = I >> 4, m = (I >> 2) & 3, q = I & 3;
        f32x4 o = {ard<AIDX(ai, 0, m, 0) + q>(), ard<AIDX(ai, 0, m, 1) + q>(), ard<AIDX(ai, 1, m, 0) + q>(), ard<AIDX(ai, 1, m, 1) + q>()};
        if (ADD) o += z[I & 15];
        asm volatile("" : "+v"(cb));
        __builtin_amdgcn_raw_buffer_store_b128(__builtin_bit_cast(u32x4, o), rC, cb + (unsigned)(ai * 128 + m * 16 + q) * pitch, 0, AUXC);
        stores<I + 1, I1, AUXC, ADD>(rC, cb, pitch, z);
    }
}
template <int I> __device__ __forceinline__ void rescale(int e) {
    if constexpr (I < 128) { float t; asm volatile("v_accvgpr_read_b32 %0, a%c2\n\tv_ldexp_f32 %0, %0, %1\n\tv_accvgpr_write_b32 a%c2, %0" : "=&v"(t) : "v"(e), "i"(I)); rescale<I + 1>(e); }
}
template <int I, int I1> __device__ __forceinline__ void ploads(rsrc_t rP, unsigned pb, unsigned pitch, f32x4 (&z)[16]) {
#pragma unroll
    for (int i = I; i < I1; ++i) z[i & 15] = __builtin_bit_cast(f32x4, __builtin_amdgcn_raw_buffer_load_b128(rP, pb, ((i >> 4) * 128 + ((i >> 2) & 3) * 16 + (i & 3)) * pitch, AUX_SYS));
}
template <int R, int O> __device__ __forceinline__ void ald(unsigned v, const void* b) { asm volatile("global_load_dwordx4 a[%c2:%c3], %0, %1 offset:%c4 sc0 sc1" :: "v"(v), "s"(b), "i"(R), "i"(R + 3), "i"(O) : "memory"); }
template <int R, int O> __device__ __forceinline__ void ast7(unsigned v, unsigned char* const (&b)[8]) {
#define AST(n) "global_store_dwordx4 %0, a[%c8:%c9], %" #n " offset:%c10 sc0 sc1\n\t"
    asm volatile(AST(1) AST(2) AST(3) AST(4) AST(5) AST(6) AST(7) :: "v"(v), "s"(b[0]), "s"(b[1]), "s"(b[2]), "s"(b[3]), "s"(b[4]), "s"(b[5]), "s"(b[6]), "i"(R), "i"(R + 3), "i"(O) : "memory");
#undef AST
}
template <int V> struct IC { static constexpr int value = V; };
#define A10(n) "a" #n "0", "a" #n "1", "a" #n "2", "a" #n "3", "a" #n "4", "a" #n "5", "a" #n "6", "a" #n "7", "a" #n "8", "a" #n "9"
#define CLAIM_ACC() asm volatile("" ::: "a0", "a1", "a2", "a3", "a4", "a5", "a6", "a7", "a8", "a9", A10(1), A10(2), A10(3), A10(4), A10(5), A10(6), A10(7), A10(8), A10(9), A10(10), A10(11), \
    "a120", "a121", "a122", "a123", "a124", "a125", "a126", "a127")

__global__ __launch_bounds__(512, 2) void fused_gemm(Params p) {
    extern __shared__ __attribute__((aligned(16))) unsigned char shm[];
    LAS unsigned char* lds = (LAS unsigned char*)shm;
    volatile LAS unsigned* const sm = (volatile LAS unsigned*)(lds + 8 * HTB);
  for (int exw = 0;;) {
    int tid = threadIdx.x, bid = blockIdx.x, rank = p.rank;
    asm volatile("" : "+v"(tid), "+s"(bid), "+s"(rank));
    const int wid = __builtin_amdgcn_readfirstlane(tid >> 6), lane = tid & 63, x = bid & 7, c = bid >> 3;
    unsigned* ctrl = (unsigned*)p.ws[rank];
    const unsigned long long tstart = __builtin_amdgcn_s_memrealtime();
    const unsigned done = __hip_atomic_load(ctrl + W_DONE, __ATOMIC_RELAXED, __HIP_MEMORY_SCOPE_AGENT);
    const int g = c - ((x & 1) ? PUSH_O : PUSH_E);
    const int q0 = g < 0 ? (x >> 1) * (PUSH_E + PUSH_O) + ((x & 1) ? PUSH_E : 0) + c : PRE_WGS - 1 - (g < GO ? g * 8 + x : GO * 8 + (g - GO) * 4 + (x >> 1));
    const int b0 = blk_at(p, q0);
    f32x4 cv[8]; float amax[2] = {0.f, 0.f};
    if (tid < 8) sm[128 + tid] = tid == 0 ? NKT : 0;
    if (tid < 4) sm[64 + tid] = 0;
    conv_load(p, b0 >> 3, b0 & 7, wid, lane, cv);
    const unsigned call = __builtin_amdgcn_readfirstlane(done) / GRID + 1;
    const unsigned par = call & 1, ncp = (call + 1) >> 1;
    const size_t bufoff = CTRL_BYTES + (size_t)par * BUF_BYTES;
    const int pw = W_PAR + par * PAR_WORDS;
    conv_store(p, b0 >> 3, b0 & 7, wid, rank, bufoff, g < 0 ? 0 : q0 < NDIR ? 0xff : 1 << rank, rank + q0 * 3 + wid * 2 + (wid >> 2), lane, cv, exw, amax);
    if (bid == 0 && tid == 0) __hip_atomic_store(ctrl + W_PAR + (par ^ 1) * PAR_WORDS + P_TICKET, 0u, __ATOMIC_RELAXED, __HIP_MEMORY_SCOPE_AGENT);
    unsigned char* sigb = p.ws[0];
#pragma unroll
    for (int d = 1; d < NR; ++d) sigb = lane == d ? p.ws[d] : sigb;
    unsigned* const sig = (unsigned*)sigb + pw;
#define GATE(word, thr, need, slow) do { if (!wait_mask(word, thr, need, ctrl + W_TMO, call, slow) && lane == 0) p.out[rank][0] = __builtin_nanf(""); } while (0)

    if (g >= 0) {
        WAIT_V(0); WAIT_L(0); BAR;
        if (tid < NR && (q0 < NDIR || tid == rank)) __hip_atomic_fetch_add(sig + P_CNT + (b0 >> 3) * 32 + (b0 & 7), 16u, __ATOMIC_RELAXED, __HIP_MEMORY_SCOPE_SYSTEM);
        if (tid == 64 && q0 >= NDIR) __hip_atomic_fetch_add(ctrl + pw + (q0 < NDIR + 32 ? P_PRE : P_PRE + 32), 1u, __ATOMIC_RELAXED, __HIP_MEMORY_SCOPE_SYSTEM);
        const int qi = g >= ((x & 1) ? GO - R0O : GE - R0E) ? NBLK : (((x & 1) ? GE - R0E : 0) + g) * 4 + (x >> 1);
        for (int q = qi; q < 128; q += 4 * (RJ - R0)) {
            const int it = 16 + NPN + (q & 15), sl = q >> 4;
            conv_load(p, it, sl, wid, lane, cv); conv_store(p, it, sl, wid, rank, bufoff, 1 << rank, rank, lane, cv, exw, amax);
            WAIT_V(0); BAR;
            if (tid == 0 || (tid == 1 && (q & 7) == rank)) __hip_atomic_fetch_add(ctrl + pw + (tid ? P_CNT + it * 32 + sl : P_SPC), tid ? 16u : 1u, __ATOMIC_RELAXED, __HIP_MEMORY_SCOPE_SYSTEM);
        }
        int word = P_PRE + 64, pass = 1;
        for (int q = PRE_WGS + qi; q < NBLK; q += 4 * (RJ - R0)) {
            const int b = blk_at(p, q);
            if (b & 0x8000) continue;
            while (__builtin_amdgcn_s_memrealtime() - tstart < (unsigned)(pass * CONV_GAP + (g & 7) * (CONV_GAP / 8))) __builtin_amdgcn_s_sleep(8);
            conv_load(p, b >> 3, b & 7, wid, lane, cv);
            conv_store(p, b >> 3, b & 7, wid, rank, bufoff, 1 << rank, rank, lane, cv, exw, amax);
            word = lane == pass ? P_CNT + (b >> 3) * 32 + (b & 7) : word;
            ++pass;
        }
#pragma unroll
        for (int o = 32; o >= 1; o >>= 1) { amax[0] = fmaxf(amax[0], __shfl_xor(amax[0], o)); amax[1] = fmaxf(amax[1], __shfl_xor(amax[1], o)); }
        if (lane < 2) __hip_atomic_fetch_max((LAS unsigned*)sm + 130 + lane, __builtin_bit_cast(unsigned, lane ? amax[1] : amax[0]), __ATOMIC_RELAXED, __HIP_MEMORY_SCOPE_WORKGROUP);
        WAIT_V(0); WAIT_L(0); BAR;
        if (tid < 2) __hip_atomic_fetch_max((unsigned long long*)(ctrl + W_AMAX) + tid, ((unsigned long long)call << 32) | sm[130 + tid], __ATOMIC_RELAXED, __HIP_MEMORY_SCOPE_AGENT);
        WAIT_V(0);
        if (tid < pass) __hip_atomic_fetch_add(ctrl + pw + word, lane ? 16u : 1u, __ATOMIC_RELAXED, __HIP_MEMORY_SCOPE_SYSTEM);
    } else {
        if (wid < PUSH_WAVES) {
            unsigned* const ticket = ctrl + pw + P_TICKET;
            const unsigned long long peers = 0xffu & ~(1u << rank);
            unsigned long long ex;
            const int rot = wid + q0 * PUSH_WAVES;
            unsigned char* bw[8]; unsigned char* bo[8];
#pragma unroll
            for (int q = 0; q < 8; ++q) { const int d = q == 7 ? rank : (rank + 1 + (q + rot) % 7) & 7; bw[q] = p.ws[d] + bufoff; bo[q] = (unsigned char*)p.out[d]; }
            CLAIM_ACC();
            asm volatile("v_accvgpr_write_b32 a1, 1");
#define BUMP(word, n) asm volatile("s_mov_b64 %0, exec\n\ts_mov_b64 exec, %3\n\tglobal_atomic_add %1, %2, off sc1\n\ts_mov_b64 exec, %0" : "=&s"(ex) : "v"(sig + (word)), "v"((unsigned)(n)), "s"(peers) : "memory")
#define SIGNAL(w) do { if ((w) >= 0) BUMP((w) >> 4, ((w) & 15) * 2); } while (0)
#define DRAW() asm volatile("s_mov_b64 %0, exec\n\ts_mov_b64 exec, 1\n\tglobal_atomic_add a0, %1, a1, off sc0\n\ts_mov_b64 exec, %0" : "=&s"(ex) : "v"(ticket) : "memory")
            DRAW();
            int stage = 0;
            int left = 0, nfwd = 0, tword = -1, word[3], sent[3] = {-1, -1, -1};
            unsigned off[3], noff = 0;
            int kind = 0, uk[3];
            bool tiles_ok = false, single = false;
            auto fetch = [&](auto reg, int& rword, unsigned& roff, int& rk) __attribute__((always_inline)) {
                constexpr int R = decltype(reg)::value;
                if (left) { --left; noff += kind ? 4 * N * 4 : 4096; }
                else if (kind < 2) {
                    if (single) WAIT_V(32); else WAIT_V(63);
                    const int k = __builtin_amdgcn_readfirstlane(__builtin_bit_cast(int, ard<0>()));
                    const int u = k < T_K0 ? NDIR * 8 + k * UPT : k < T_K1 ? U_K0 + (k - T_K0) * WUPT : k < T_K2 ? U_K1 + (k - T_K1) * UPT : U_K2 + k - T_K2;
                    if ((unsigned)u >= (unsigned)U_END) { kind = 2; tword = -1; WAIT_V(0); }
                    else {
                        left = k < T_K0 || (k >= T_K1 && k < T_K2) ? UPT - 1 : k < T_K1 ? WUPT - 1 : 0;
                        single = left == 0;
                        DRAW();
                        const int b = blk_at(p, u >> 3), fv = (b & 0xff) * 8 + (u & 7), st = u >= PRE_WGS * 8 ? 3 : u >= (NDIR + 32) * 8 ? 2 : 1;
                        if (st > stage) {
                            GATE(ctrl + pw + P_PRE + (st - 1) * 32, (st == 3 ? PRE_WGS : st == 2 ? PRE_WGS - 32 - NDIR : 32) * ncp, 1, 0);
                            stage = st;
                        }
                        kind = b >> 15;
                        if (kind && !tiles_ok) { GATE(ctrl + pw + P_TILE, TILE_PER_CALL * ncp, 1, 0); tiles_ok = true; }
                        tword = kind ? -1 : ((P_CNT + (b >> 3) * 32 + (b & 7)) << 4) + left + 1;
                        nfwd += kind ? left + 1 : 0;
                        noff = kind ? (unsigned)(fv & 1023) * 4u * (N * 4) + (NPN + (fv >> 10) * 8 + rank) * 1024 : (b >> 3) * PANEL + ((b & 7) * 8 + rank) * KTB + (u & 7) * 4096;
                    }
                }
                const unsigned vn = noff + lane * 16;
                asm volatile("s_nop 4" :: "s"(bw[7]), "s"(bo[7]));
                if (kind == 0) { ald<R, 0>(vn, bw[7]); ald<R + 4, 1024>(vn, bw[7]); ald<R + 8, 2048>(vn, bw[7]); ald<R + 12, 3072>(vn, bw[7]); }
                if (kind == 1) { ald<R, 0>(vn, bo[7]); ald<R + 4, 0>(vn + N * 4, bo[7]); ald<R + 8, 0>(vn + 2 * N * 4, bo[7]); ald<R + 12, 0>(vn + 3 * N * 4, bo[7]); }
                rword = left ? -1 : tword; roff = noff; rk = kind;
            };
            auto step = [&](auto reg, int& rword, int& rsent, unsigned& roff, int& rk) __attribute__((always_inline)) {
                constexpr int R = decltype(reg)::value;
                if (kind < 2) { WAIT_V(63); SIGNAL(rsent); rsent = -1; }
                const unsigned vo = roff + lane * 16;
                if (rk == 0) { asm volatile("s_nop 4" :: "s"(bw[0]), "s"(bw[1]), "s"(bw[2]), "s"(bw[3]), "s"(bw[4]), "s"(bw[5]), "s"(bw[6]));
                               ast7<R, 0>(vo, bw); ast7<R + 4, 1024>(vo, bw); ast7<R + 8, 2048>(vo, bw); ast7<R + 12, 3072>(vo, bw); }
                if (rk == 1) { asm volatile("s_nop 4" :: "s"(bo[0]), "s"(bo[1]), "s"(bo[2]), "s"(bo[3]), "s"(bo[4]), "s"(bo[5]), "s"(bo[6]));
                               ast7<R, 0>(vo, bo); ast7<R + 4, 0>(vo + N * 4, bo); ast7<R + 8, 0>(vo + 2 * N * 4, bo); ast7<R + 12, 0>(vo + 3 * N * 4, bo); }
                if (rk < 2) { SIGNAL(rsent); rsent = rword; }
                fetch(reg, rword, roff, rk);
            };
            WAIT_V(0); fetch(IC<4>(), word[0], off[0], uk[0]);
            if (!left) WAIT_V(0);
            fetch(IC<20>(), word[1], off[1], uk[1]);
            if (!left) WAIT_V(0);
            fetch(IC<36>(), word[2], off[2], uk[2]);
            WAIT_V(0);
            do {
                step(IC<4>(), word[0], sent[0], off[0], uk[0]);
                step(IC<20>(), word[1], sent[1], off[1], uk[1]);
                step(IC<36>(), word[2], sent[2], off[2], uk[2]);
            } while (uk[0] + uk[1] < 4);
            WAIT_V(0);
            SIGNAL(sent[0]); SIGNAL(sent[1]); SIGNAL(sent[2]);
            BUMP(P_FWD + rank * 32, nfwd);
        }
        if ((wid == 2 || wid == 3) && q0 < 16) {
            static_assert(NPUSH >= 16 && PUSH_WAVES <= 2, "one pusher per (panel set, unit)");
            const rsrc_t rl = mk(p.ws[rank] + bufoff, BUF_BYTES);
            const int it = 16 + NPN + (q0 & 8);
            GATE(ctrl + pw + P_SPC, 128 * ncp, 1, 0);
            for (int sl = 0; sl < 8; ++sl) {
                const unsigned off = it * PANEL + (sl * 8 + rank) * KTB + (q0 & 7) * 4096 + (wid & 1) * 2048 + lane * 16;
                u32x4 z[7][2];
#pragma unroll
                for (int q = 0; q < 7; ++q)
#pragma unroll
                    for (int i = 0; i < 2; ++i) z[q][i] = __builtin_amdgcn_raw_buffer_load_b128(rl, off + i * 1024, ((rank + 1 + (q + q0) % 7) & 7) * PANEL, AUX_SYS);
                GATE(ctrl + pw + P_TICKET, (sl * 16 + q0) * T_K0 / 128, 1, 1);
#pragma unroll
                for (int i = 0; i < 2; ++i)
#pragma unroll
                    for (int q = 0; q < 7; ++q) { const int d = (rank + 1 + (q + q0) % 7) & 7; __builtin_amdgcn_raw_buffer_store_b128(z[q][i], mk(p.ws[d] + bufoff + d * PANEL, BUF_BYTES - 7 * PANEL), off + i * 1024, 0, AUX_SYS); }
                WAIT_V(0);
                if (lane < NR && lane != rank) __hip_atomic_fetch_add(sig + P_CNT + (it + lane) * 32 + sl, 1u, __ATOMIC_RELAXED, __HIP_MEMORY_SCOPE_SYSTEM);
            }
        }
    }
    if (bid == 1 && wid == 4) {
        GATE(ctrl + pw + P_PRE + 64, PRE_WGS * ncp, 1, 1);
        const unsigned long long m0 = __hip_atomic_load((unsigned long long*)(ctrl + W_AMAX), __ATOMIC_RELAXED, __HIP_MEMORY_SCOPE_AGENT);
        const unsigned long long m1 = __hip_atomic_load((unsigned long long*)(ctrl + W_AMAX) + 1, __ATOMIC_RELAXED, __HIP_MEMORY_SCOPE_AGENT);
        unsigned long long* const ain = (unsigned long long*)((unsigned*)sigb + W_AIN + par * 64) + rank * 2;
        if (lane < NR) { __hip_atomic_store(ain, m0, __ATOMIC_RELAXED, __HIP_MEMORY_SCOPE_SYSTEM); __hip_atomic_store(ain + 1, m1, __ATOMIC_RELAXED, __HIP_MEMORY_SCOPE_SYSTEM); }
        const bool ok = wait_mask(ctrl + W_AIN + par * 64 + (lane & 15) * 2 + 1, call, 0xffff, ctrl + W_TMO, call, 1) != 0;
        if (!ok && lane == 0) p.out[rank][0] = __builtin_nanf("");
        int m = __hip_atomic_load(ctrl + W_AIN + par * 64 + (lane & 15) * 2, __ATOMIC_RELAXED, __HIP_MEMORY_SCOPE_SYSTEM);
#pragma unroll
        for (int o = 8; o >= 2; o >>= 1) m = max(m, __shfl_xor(m, o));
        const int be = m >> 23, e = !ok || m == 0 || be == 255 || (be >= 127 - 8 && be < 127 + 15) ? 0 : 127 + 13 - (be ? be : 1);
        const unsigned dec = ((unsigned)__builtin_amdgcn_readlane(e, 0) << 16) | ((unsigned)__builtin_amdgcn_readlane(e, 1) & 0xffffu);
        if (lane == 0) __hip_atomic_store((unsigned long long*)(ctrl + W_DEC), ((unsigned long long)call << 32) | dec, __ATOMIC_RELAXED, __HIP_MEMORY_SCOPE_AGENT);
    }
    if (bid == 0 && wid == 4) {
        unsigned long long* const tab = (unsigned long long*)(ctrl + W_ETA);
        LAS unsigned* const kb = (LAS unsigned*)lds; LAS unsigned* const ord = kb + GRID;
        unsigned key[4]; bool all = false;
        while (!all && __builtin_amdgcn_readfirstlane(__hip_atomic_load(ctrl + W_DONE, __ATOMIC_RELAXED, __HIP_MEMORY_SCOPE_AGENT)) % GRID < NPUSH
               && __builtin_amdgcn_s_memrealtime() - tstart < (1u << 20)) {
            __builtin_amdgcn_s_sleep(127); __builtin_amdgcn_s_sleep(127);
            int n = 0;
            const unsigned now = (unsigned)(__builtin_amdgcn_s_memrealtime() - tstart);
#pragma unroll
            for (int i = 0; i < 4; ++i) {
                const unsigned d = (unsigned)__hip_atomic_load(tab + i * 64 + lane, __ATOMIC_RELAXED, __HIP_MEMORY_SCOPE_AGENT) - (unsigned)tstart;
                key[i] = d > now && d < (1u << 21) && ((i * 64 + lane) >> 3) >= ((lane & 1) ? PUSH_O : PUSH_E) ? (d << 8) + i * 64 + lane : 0u;
                n += __popcll(__ballot(key[i] != 0));
            }
            all = n == PRE_WGS;
        }
        if (all) {
            int r[4] = {0, 0, 0, 0};
#pragma unroll
            for (int i = 0; i < 4; ++i) kb[i * 64 + lane] = key[i];
            for (int k = 0; k < GRID; ++k) { const unsigned o = kb[k];
#pragma unroll
                for (int i = 0; i < 4; ++i) r[i] += o < key[i]; }
#pragma unroll
            for (int i = 0; i < 4; ++i) if (key[i]) ord[r[i] - NPUSH] = key[i];
            const unsigned cap = (unsigned)(__builtin_amdgcn_s_memrealtime() - tstart) + 30000u, first = (ord[0] >> 8) < cap ? ord[0] >> 8 : cap;
            while (__builtin_amdgcn_s_memrealtime() - tstart + STEAL_LEAD < first) __builtin_amdgcn_s_sleep(32);
            for (int k = lane; k < PRE_WGS / 2; k += 64) {
                const int th = ord[k] & 0xff, v = ord[PRE_WGS - 1 - k] & 0xff;
                const unsigned long long et = __hip_atomic_load(tab + th, __ATOMIC_RELAXED, __HIP_MEMORY_SCOPE_AGENT), ev = __hip_atomic_load(tab + v, __ATOMIC_RELAXED, __HIP_MEMORY_SCOPE_AGENT);
                const int rem = (int)((unsigned)ev + (unsigned)(ev >> 48) - (unsigned)et - (unsigned)(et >> 48)) - STEAL_OVH;
                int n = rem > 0 ? rem / (int)(((ev >> 32) & 0xffff) + ((et >> 32) & 0xffff) + 1) : 0;
                if (n > NKT / 4) n = NKT / 4;
                if (n >= STEAL_MIN)
                    __hip_atomic_store((unsigned long long*)(ctrl + W_BOX + th * 32 + 2), ((unsigned long long)call << 32) | (call << 16) | (v << 8) | (NKT - 2 * n), __ATOMIC_RELAXED, __HIP_MEMORY_SCOPE_AGENT);
            }
        }
    }
    if (bid == 1 && wid == 0) {
        GATE(ctrl + pw + P_FWD + (lane & 7) * 32, FWD_PER_CALL * ncp, 0xff & ~(1 << rank), 0);
        __builtin_amdgcn_fence(__ATOMIC_ACQUIRE, "");
    }
    if (g >= 0) {
        const int wr = wid >> 2, wc = wid & 3, fr = lane & 15, fq = lane >> 4;
        const rsrc_t rAB = mk(p.ws[rank] + bufoff, BUF_BYTES), rCt = mk(ctrl, (unsigned)CTRL_BYTES);
        const unsigned voff = wid * 2048 + lane * 16, ldsw = wid * 2048;
        LAS unsigned char* pa = lds + wr * 8192 + lane * 16; LAS unsigned char* pb = lds + SB(0, 0) + wc * 4096 + lane * 16;
        asm volatile("" : "+v"(pa), "+v"(pb));
        const unsigned thr = CNT_PER_CALL * ncp;
        int j = g >= ((x & 1) ? GO - R0O : GE - R0E) ? 0 : 1, t0 = 0; Tile cur, nxt;
        slot(j, g, x, rank, cur);
        bool bad = false, helping = false;
        unsigned tjob = (unsigned)tstart, pace = 0, eta6 = 0;
        unsigned have = 0;
        if (wid == 0) {
            const int ln = (lane & 8) ? cur.pm : 16 + cur.pn;
            have = (unsigned)wait_mask(ctrl + pw + P_CNT + ln * 32 + (lane & 7), thr, 0x101, ctrl + W_TMO, call, 1) & 0xffffu;
            bad = have == 0;
        }
        asm volatile("s_waitcnt lgkmcnt(0)" ::: "memory"); BAR; asm volatile("" ::: "memory");
        CLAIM_ACC();
        h8 At[4][2], B0[2][2], B1[2][2];
        unsigned cA = cur.pm * PANEL, cB = A_BYTES + cur.pn * PANEL;
        STAGE(SB(0, 0), cB); STAGE(SB(0, 1), cB + HTB); STAGE(SA(0, 0), cA); STAGE(SA(0, 1), cA + HTB);
        if (wr == 1) BAR;
        WAIT_V(2); BAR;
        STAGE(SB(1, 0), cB + KTB); STAGE(SA(1, 0), cA + KTB); STAGE(SB(1, 1), cB + HTB + KTB);
        WAIT_V(6); BAR;
        for (;;) {
            bool has_next = slot(j + 1, g, x, rank, nxt);
            const bool final = j == 7;
            if (!has_next) nxt = cur;
            unsigned nA = nxt.pm * PANEL, nB = A_BYTES + nxt.pn * PANEL;
            const int sel = (lane >> 3) & 3, ln = sel == 0 ? 16 + cur.pn : sel == 1 ? cur.pm : sel == 2 ? 16 + nxt.pn : nxt.pm;
            unsigned fw = pw + P_CNT + ln * 32 + (lane & 7), need_next = 0x01010000u;
            if (wid == 0) {
                const unsigned now = (unsigned)__builtin_amdgcn_s_memrealtime(), dur = now - tjob;
                tjob = now; pace = dur / (NKT / 2);
                if (j == 6) eta6 = now + 2 * dur;
                const unsigned corr = (final ? now + dur - eta6 : 0u) + 0x8000u;
                if ((j == 6 || final) && lane == 0) __hip_atomic_store((unsigned long long*)(ctrl + W_ETA) + bid,
                    ((unsigned long long)((corr << 16) | (pace & 0xffffu)) << 32) | eta6, __ATOMIC_RELAXED, __HIP_MEMORY_SCOPE_AGENT);
                if (final) __builtin_amdgcn_raw_ptr_buffer_load_lds(rCt, (LAS void*)(lds + 8 * HTB + 768), 4, lane == 2 ? W_TMO * 4 : (W_DEC + (lane & 1)) * 4, 0, 0, AUX_SYS);
                __builtin_amdgcn_s_waitcnt(0xC07F);
            }
            int plan = NKT;
            auto arrival = [&](int t) __attribute__((always_inline)) {
                if (wid == 0 && (t & 4)) {
                    const unsigned need = t >= NKT - 4 ? need_next : 0x202u << (t >> 3);
                    if ((have & need) != need && !helping) {
                        if (!(t & 2)) __builtin_amdgcn_raw_ptr_buffer_load_lds(rCt, (LAS void*)(lds + 8 * HTB), 4, fw * 4, 0, 0, AUX_SYS);
                        else {
                            have = (unsigned)__ballot(sm[lane] >= thr);
                            if ((have & need) != need) {
                                have = (unsigned)wait_mask(ctrl + fw, thr, need, ctrl + W_TMO, call, j >= 2 ? LAZY : 1);
                                bad |= have == 0;
                            }
                            asm volatile("" ::: "memory");
                        }
                    }
                }
            };
#define TRIP(Z, HOOK) do { \
                const unsigned a1 = cA + (t + 1) * KTB; \
                const unsigned a2 = t == NKT - 2 ? nA : cA + (t + 2) * KTB, b2 = t == NKT - 2 ? nB : cB + (t + 2) * KTB; \
                const unsigned a3 = a2 + KTB, b3 = b2 + KTB; \
                LDB(B0, 0, 0); LDB(B1, 0, 1); SCHED; LDA(At, 0, 0); STAGE(SA(1, 1), a1 + HTB); \
                WAIT_V(8); WAIT_L0; BAR; MMA2(0, Z); BAR; SCHED; \
                LDA(At, 0, 1); STAGE(SB(0, 0), b2); STAGE(SB(0, 1), b2 + HTB); STAGE(SA(0, 0), a2); \
                WAIT_V(8); WAIT_L0; BAR; MMA2(1, Z); BAR; SCHED; \
                LDB(B0, 1, 0); LDB(B1, 1, 1); SCHED; LDA(At, 1, 0); STAGE(SA(0, 1), a2 + HTB); \
                HOOK \
                WAIT_V(8); WAIT_L0; BAR; MMA2(0, false); BAR; SCHED; \
                LDA(At, 1, 1); STAGE(SB(1, 0), b3); STAGE(SB(1, 1), b3 + HTB); STAGE(SA(1, 0), a3); \
                WAIT_V(8); WAIT_L0; BAR; MMA2(1, false); BAR; SCHED; } while (0)
            volatile LAS unsigned* pvp = sm + (lane < 4 ? 64 + lane : 128);
            int t = t0;
            TRIP(true, ); t += 2;
            for (const int te = final ? NKT / 2 - 12 : NKT; t < te; t += 2) { arrival(t); TRIP(false, ); }
            for (; t < (plan & 0xff); t += 2) {
                if (t == NKT - 2 && (plan >> 16)) {
                    const int v = (plan >> 8) & 0xff;
                    slot(7, (v >> 3) - ((v & 1) ? PUSH_O : PUSH_E), v & 7, rank, nxt);
                    has_next = true; nA = nxt.pm * PANEL + (plan >> 16) * KTB; nB = A_BYTES + nxt.pn * PANEL + (plan >> 16) * KTB;
                }
                if (t == NKT - 4 && need_next != 0x01010000u) have &= 0xffffu;
                arrival(t);
                unsigned pv = 0;
                const bool look = !(t & 2) && plan == NKT;
                if (look) pv = *pvp;
                auto hook = [&]() __attribute__((always_inline)) {
                    if (!look) return;
                    plan = __builtin_amdgcn_readlane(pv, 4);
                    if (wid == 0 && plan == NKT) {
                        const unsigned of = __builtin_amdgcn_readlane(pv, 0), as = __builtin_amdgcn_readlane(pv, 2), tag = call & 0xffffu;
                        if (__builtin_amdgcn_readlane(pv, 1) == call && (of >> 16) == tag && (int)(of & 0xff) >= t + 6) { if (lane == 0) sm[128] = of & 0xffff; }
                        else if (__builtin_amdgcn_readlane(pv, 3) == call && (as >> 16) == tag && t <= NKT - 8) {
                            const int v = (as >> 8) & 0xff;
                            if (lane == 0) {
                                sm[128] = ((as & 0xff) << 16) | (v << 8) | NKT;
                                __hip_atomic_store((unsigned long long*)(ctrl + W_BOX + v * 32), ((unsigned long long)call << 32) | (tag << 16) | (bid << 8) | (as & 0xff), __ATOMIC_RELAXED, __HIP_MEMORY_SCOPE_AGENT);
                            }
                            Tile vt; slot(7, (v >> 3) - ((v & 1) ? PUSH_O : PUSH_E), v & 7, rank, vt);
                            if (lane & 16) fw = pw + P_CNT + ((lane & 8) ? vt.pm : 16 + vt.pn) * 32 + (lane & 7);
                            have &= 0xffffu; need_next = 0xffff0000u;
                        }
                        __builtin_amdgcn_raw_ptr_buffer_load_lds(rCt, (LAS void*)(lds + 8 * HTB + 256), 4, (lane & 3) * 4, (W_BOX + bid * 32) * 4, 0, AUX_SYS);
                    }
                };
                TRIP(false, hook(););
            }
            if (wr == 0) BAR;
            const rsrc_t rS = mk(p.ws[rank] + SCR_OFF, SCR_BYTES);
            const unsigned sb = (unsigned)(wr * 64 + 4 * fq) * 1024 + (wc * 64 + 4 * fr) * 4;
            const bool helped = (plan & 0xff) < NKT;
            if (helped) {
                if (wid == 0) { bad |= !wait_mask(ctrl + W_BOX + ((plan >> 8) & 0xff) * 32 + 4, call, 1, ctrl + W_TMO, call); }
                asm volatile("" ::: "memory"); BAR; asm volatile("" ::: "memory");
            }
            asm volatile("s_nop 7" ::: "memory");
            if (final && wid == 0 && __builtin_amdgcn_readfirstlane(sm[194]) == call) bad = true;
            if (exw) rescale<0>(-((exw >> 16) + (short)exw));
            if (bad) asm volatile("v_accvgpr_write_b32 a0, %0" :: "v"(__builtin_nanf("")));
#define C_AUX AUX_NT
            const bool own = cur.pn >= NPN || helping;
            const rsrc_t rC = helping ? rS : mk(p.out[rank], OUT_BYTES);
            const unsigned pitch = helping ? 1024 : N * 4;
            const unsigned cb = helping ? bid * 262144u + sb : (unsigned)(cur.pm * 256 + wr * 64 + 4 * fq) * (N * 4) + cur.pn * 1024 + (wc * 64 + 4 * fr) * 4;
            const unsigned pbase = ((plan >> 8) & 0xff) * 262144u + sb;
            f32x4 z[16];
            if (own) stores<0, 32, AUX_SYS, false>(rC, cb, pitch, z);
            else if (!helped) stores<0, 32, C_AUX, false>(rC, cb, N * 4, z);
            else {
                ploads<0, 16>(rS, pbase, 1024, z);
                stores<0, 8, C_AUX, true>(rC, cb, N * 4, z); ploads<16, 24>(rS, pbase, 1024, z);
                stores<8, 16, C_AUX, true>(rC, cb, N * 4, z); ploads<24, 32>(rS, pbase, 1024, z);
                stores<16, 32, C_AUX, true>(rC, cb, N * 4, z);
            }
            if (cur.pn >= NPN && !helping) {
                WAIT_V(0);
                if (lane == 0) __hip_atomic_fetch_add(ctrl + pw + P_TILE, 1u, __ATOMIC_RELAXED, __HIP_MEMORY_SCOPE_AGENT);
            }
            if (!has_next) break;
            t0 = plan >> 16; helping = final;
            cur = nxt; cA = nA - t0 * KTB; cB = nB - t0 * KTB; ++j; have >>= 16;
            if (wr == 1) BAR;
        }
        WAIT_V(0);
        if (helping) {
            BAR;
            if (tid == 0) __hip_atomic_store(ctrl + W_BOX + bid * 32 + 4, call, __ATOMIC_RELAXED, __HIP_MEMORY_SCOPE_AGENT);
        }
    }
    if (wid == (g < 0 ? 5 : 0) && (g < 0 || __builtin_amdgcn_readfirstlane(sm[193]) != call)) {
        const bool ok = wait_mask(ctrl + W_DEC + 1, call, 1, ctrl + W_TMO, call, 1) != 0;
        const unsigned dec = __hip_atomic_load(ctrl + W_DEC, __ATOMIC_RELAXED, __HIP_MEMORY_SCOPE_AGENT);
        if (lane == 0) { sm[192] = ok ? dec : 0u; if (!ok) p.out[rank][0] = __builtin_nanf(""); }
    }
    WAIT_L(0); BAR;
    if (tid == 0) __hip_atomic_fetch_add(ctrl + W_DONE, 1u, __ATOMIC_RELAXED, __HIP_MEMORY_SCOPE_AGENT);
    const int dec = __builtin_amdgcn_readfirstlane(sm[192]);
    if (exw || !dec) break;
    exw = dec;
    if (wid == 0 && !wait_mask(ctrl + W_DONE, call * GRID, 1, ctrl + W_TMO, call, 1) && lane == 0) sm[192] = 0;
    WAIT_V(0); WAIT_L(0); BAR;
    if (__builtin_amdgcn_readfirstlane(sm[192]) == 0) { if (tid == 0) p.out[rank][0] = __builtin_nanf(""); break; }
    BAR;
  }
}

void make_order(Params& p) {
    double key[NBLK]; int val[NBLK], n = 0;
    const double P0 = (14 + R0 / 4.0 + 2) * 0.575, P1 = 10.3;
    auto add = [&](double k, int v) { int i = n++; while (i > 0 && key[i - 1] > k) { key[i] = key[i - 1]; val[i] = val[i - 1]; --i; } key[i] = k; val[i] = v; };
    for (int sl = 0; sl < 8; ++sl) for (int a = 0; a < 16; ++a) add(sl * P0 + a * 0.01 - XLEAD, a * 8 + sl);
    for (int j = 0; j < 8; ++j) {
        const int e0 = j == 0 ? 0 : R0 + (j - 1) * RJ, ne = j == 0 ? R0E : GE, no = j == 0 ? R0O : GO;
        for (int col = (e0 + 3) / 4; col * 4 < e0 + ne + no; ++col) {
            if (col == SPLIT_COL || col == SPLIT_COL + 1) continue;
            const int e = col * 4 - e0;
            double k = e < ne ? (e + GE - ne + 0.5) / GE : (double)(e - ne + GO - no) / GO;
            if (e < ne && e + 3 >= ne) k = (double)(GO - no) / GO;
            for (int sl = 0; sl < 8; ++sl)
                add((j == 0 ? sl * P0 + P0 - P1 : 8 * P0 + ((j - 1) * 8 + sl) * P1) + 0.5 * P1 + (k - 0.5) * SKEW, (16 + col - (col > SPLIT_COL ? 2 : 0)) * 8 + sl);
        }
    }
    for (int f = 0; f < (int)FWD_PER_CALL / 8; ++f) add(8 * P0 + (FWD_FROM - 1) * 8 * P1 + (f / FWD_RUN + 1) * FWD_RUN * P1 * 10 / FWD_MIX + f * 1e-3, 0x8000 + f);
    for (int u = 0; u < NBLK; ++u) p.blk[u >> 1] |= (unsigned)val[u] << ((u & 1) * 16);
}
}

extern "C" void kernel_launch(int rank, int nranks, ncclComm_t comm, hipStream_t stream,
                              void* const* d_in, const int* in_sizes, int n_in,
                              void* d_out, int out_size, const int* in_dtypes, int out_dtype,
                              void* const* d_out_all, void* const* d_ws_all, size_t ws_bytes) {
    if (nranks != NR || (unsigned)rank >= (unsigned)NR || n_in != 2 || in_sizes[0] != M * KL || in_sizes[1] != KL * N
        || out_size != M * N || ws_bytes < WS_NEED) { fprintf(stderr, "unexpected problem shape\n"); return; }
    static bool init = false;
    if (!init) { HIP_CHECK(hipFuncSetAttribute((const void*)fused_gemm, hipFuncAttributeMaxDynamicSharedMemorySize, LDS_BYTES)); init = true; }
    Params p = {};
    p.x = (const float*)d_in[0]; p.w = (const float*)d_in[1]; p.rank = rank;
    for (int r = 0; r < NR; ++r) { p.out[r] = (float*)d_out_all[r]; p.ws[r] = (unsigned char*)d_ws_all[r]; }
    make_order(p);
    hipLaunchKernelGGL(fused_gemm, dim3(GRID), dim3(512), LDS_BYTES, stream, p);
    HIP_CHECK(hipGetLastError());
}
```
